```python
import numpy as np

kernel_name = "gpu_gemm_rs_m4096_k4096_n4096_e4m3e5m2_dqsilu"


ABSMAX_RTOL = 0.05

COMPUTE_DTYPE = "float32"
EPILOGUE = "dequant_silu"
IN_QUANT = None
OUT_QUANT = None
IN_FP8 = "e4m3e5m2"
IN_DTYPES = {"x": "e4m3", "w_mat": "e5m2",
             "scale_x": "float32", "scale_w": "float32"}
WORLD_SIZE = 8

def _snap_fp8(x, mant_bits, max_abs, min_normal):
    a = np.abs(x.astype(np.float64))
    _m, _e = np.frexp(a)
    q = np.where(a >= min_normal, np.ldexp(1.0, _e - 1 - mant_bits),
                 min_normal * 2.0 ** -mant_bits)
    snapped = np.minimum(np.round(a / q) * q, max_abs)
    return (np.sign(x) * snapped).astype(np.float32)

def setup_inputs():

    x = _snap_fp8(np.random.randn(4096, 4096).astype(np.float32), 3, 448.0, 2.0 ** -6)
    w_mat = _snap_fp8(np.random.randn(4096, 4096).astype(np.float32), 2, 57344.0, 2.0 ** -14)
    scale_x = (0.02 * (1.0 + np.random.rand(1))).astype(np.float32)
    scale_w = (0.02 * (1.0 + np.random.rand(1))).astype(np.float32)
    return {"x": x, "w_mat": w_mat, "scale_x": scale_x, "scale_w": scale_w}

def reference(x, w_mat, scale_x, scale_w):
    acc = (x @ w_mat).astype(np.float32)
    y = (acc * (scale_x * scale_w)).astype(np.float32)
    return (y / (1.0 + np.exp(-np.clip(y, -60.0, 60.0)))).astype(np.float32)
```

```cpp
#include <hip/hip_runtime.h>
#include <rccl/rccl.h>
#include <cstdio>
#include <cstdint>

#define HIP_CHECK(expr) do { hipError_t _e = (expr); if (_e != hipSuccess) { \
    fprintf(stderr, "HIP error %s at %s:%d\n", hipGetErrorString(_e), __FILE__, __LINE__); \
    return; } } while (0)

typedef int      v8i  __attribute__((ext_vector_type(8)));
typedef unsigned v4u  __attribute__((ext_vector_type(4)));
typedef float    v16f __attribute__((ext_vector_type(16)));

namespace {

constexpr int NR = 8;
constexpr int NN = 4096;
constexpr int MS = 512;
constexpr int KS = 512;
constexpr int NWG = 256;
constexpr int NTHREADS = 320;

constexpr size_t FLAG_OFF  = 0;
constexpr size_t TMO_OFF   = 4096;
constexpr size_t EPOCH_OFF = 8192;
constexpr size_t SLOT_OFF  = 1u << 20;
constexpr size_t WBLK      = 8192;
constexpr size_t WSRC      = 8 * 32 * WBLK;
constexpr size_t XSRC      = 32 * 512 * 16;
constexpr size_t X_OFF     = NR * WSRC;
constexpr size_t SLOT_BYTES = X_OFF + NR * XSRC;
constexpr size_t WS_NEED   = SLOT_OFF + 2 * SLOT_BYTES;

constexpr unsigned long long DEADLINE_TICKS = 50000000ull;
constexpr int AUX_SYS = 17;
constexpr int RSRC3 = 0x00020000;

struct Params {
  const uint8_t* x;
  const uint8_t* w;
  const float* sx;
  const float* sw;
  float* out;
  uint8_t* ws[NR];
  int rank;
};

#define WAIT_VM(n) asm volatile("s_waitcnt vmcnt(" #n ")" ::: "memory")

__device__ __forceinline__ v4u transpose4(unsigned a0, unsigned a1, unsigned a2, unsigned a3) {
  const unsigned t0 = __builtin_amdgcn_perm(a1, a0, 0x05010400u);
  const unsigned t1 = __builtin_amdgcn_perm(a1, a0, 0x07030602u);
  const unsigned t2 = __builtin_amdgcn_perm(a3, a2, 0x05010400u);
  const unsigned t3 = __builtin_amdgcn_perm(a3, a2, 0x07030602u);
  v4u r;
  r[0] = __builtin_amdgcn_perm(t2, t0, 0x05040100u);
  r[1] = __builtin_amdgcn_perm(t2, t0, 0x07060302u);
  r[2] = __builtin_amdgcn_perm(t3, t1, 0x05040100u);
  r[3] = __builtin_amdgcn_perm(t3, t1, 0x07060302u);
  return r;
}

struct Frag {
  v8i a0, a1, b;
};

}

__global__ void __launch_bounds__(NTHREADS, 1) fused_ag_gemm(Params p) {
  const int b = blockIdx.x;
  const int wave = __builtin_amdgcn_readfirstlane(threadIdx.x >> 6);
  const int lane = threadIdx.x & 63;
  const int rank = p.rank;
  uint8_t* const myws = p.ws[rank];

  unsigned* const ep = (unsigned*)(myws + EPOCH_OFF) + b * 8 + wave;
  const unsigned call0 = __builtin_amdgcn_readfirstlane(
      __hip_atomic_load(ep, __ATOMIC_RELAXED, __HIP_MEMORY_SCOPE_AGENT));

  if (wave == 4) {
    const int d = b >> 5;
    const int i = b & 31;
    const auto rsW = __builtin_amdgcn_make_buffer_rsrc((void*)p.w, 0, KS * NN, RSRC3);
    const auto rsX = __builtin_amdgcn_make_buffer_rsrc((void*)p.x, 0, 4096 * KS, RSRC3);

    v4u xr[8];
    {
      const int off = (d * MS + (i >> 2) * 64 + lane) * KS + (i & 3) * 128;
#pragma unroll
      for (int t = 0; t < 8; ++t) xr[t] = __builtin_amdgcn_raw_buffer_load_b128(rsX, off + t * 16, 0, 0);
    }
    const int woff = ((lane >> 5) * 32) * NN + i * 128 + (lane & 31) * 4;
    unsigned wr[2][32];
#pragma unroll
    for (int t = 0; t < 32; ++t) wr[0][t] = __builtin_amdgcn_raw_buffer_load_b32(rsW, woff + t * NN, 0, 0);

    if (lane == 0) __hip_atomic_store(ep, call0 + 1, __ATOMIC_RELAXED, __HIP_MEMORY_SCOPE_AGENT);

    uint8_t* const slot = p.ws[d] + SLOT_OFF + (size_t)(call0 & 1) * SLOT_BYTES;
    const auto rdW = __builtin_amdgcn_make_buffer_rsrc((void*)(slot + (size_t)rank * WSRC), 0, (int)WSRC, RSRC3);
    const auto rdX = __builtin_amdgcn_make_buffer_rsrc((void*)(slot + X_OFF + (size_t)rank * XSRC), 0, (int)XSRC, RSRC3);
    unsigned* const flag = (unsigned*)(p.ws[d] + FLAG_OFF) + rank * 32 + i;
    const unsigned fbase = call0 * 16;

    {
      const int off = ((i & 3) * 8) * 8192 + ((i >> 2) * 64 + lane) * 16;
#pragma unroll
      for (int t = 0; t < 8; ++t) {
        __builtin_amdgcn_raw_buffer_store_b128(xr[t], rdX, off + t * 8192, 0, AUX_SYS);
        if (t >= 3) WAIT_VM(3);
      }
    }
    const int doff = i * (int)WBLK + ((lane >> 5) * 2) * 2048 + (lane & 31) * 16;
#pragma unroll
    for (int kb = 0; kb < 8; ++kb) {
#pragma unroll
      for (int j = 0; j < 4; ++j) {
        if (kb + 1 < 8) {
#pragma unroll
          for (int t = 0; t < 8; ++t)
            wr[(kb + 1) & 1][j * 8 + t] =
                __builtin_amdgcn_raw_buffer_load_b32(rsW, woff + ((kb + 1) * 64 + j * 8 + t) * NN, 0, 0);
        }
#pragma unroll
        for (int u = 0; u < 2; ++u) {
          const unsigned* r = &wr[kb & 1][j * 8 + u * 4];
          const v4u v = transpose4(r[0], r[1], r[2], r[3]);
          __builtin_amdgcn_raw_buffer_store_b128(v, rdW, kb * (int)(32 * WBLK) + doff + (2 * j + u) * 512, 0, AUX_SYS);
        }
        if (kb + 1 < 8) { WAIT_VM(20); } else { WAIT_VM(4); }
        if (j == 1) {
          if (kb + 1 < 8) { WAIT_VM(20); } else { WAIT_VM(4); }
          if (lane == 0) __hip_atomic_store(flag, fbase + 1 + kb, __ATOMIC_RELAXED, __HIP_MEMORY_SCOPE_SYSTEM);
        }
      }
    }
    WAIT_VM(0);
    if (lane == 0) __hip_atomic_store(flag, fbase + 9, __ATOMIC_RELAXED, __HIP_MEMORY_SCOPE_SYSTEM);
    return;
  }

  if (lane == 0) __hip_atomic_store(ep, call0 + 1, __ATOMIC_RELAXED, __HIP_MEMORY_SCOPE_AGENT);
  const int nblk = (b & 7) * 4 + ((b >> 3) & 3);
  const int mblk = b >> 5;
  const unsigned fbase = call0 * 16;
  const unsigned call = call0 + 1;
  unsigned* const tmo = (unsigned*)(myws + TMO_OFF);
  uint8_t* const slot = myws + SLOT_OFF + (size_t)(call0 & 1) * SLOT_BYTES;
  const auto rsW = __builtin_amdgcn_make_buffer_rsrc((void*)slot, 0, (int)X_OFF, RSRC3);
  const auto rsX = __builtin_amdgcn_make_buffer_rsrc((void*)(slot + X_OFF), 0, (int)(NR * XSRC), RSRC3);

  const int q8 = lane & 7;
  const bool isx = (q8 >= 1 && q8 <= 4);
  const unsigned* const myflag =
      (const unsigned*)(myws + FLAG_OFF) + (lane >> 3) * 32 + (isx ? (mblk * 4 + q8 - 1) : nblk);

  const int g = lane >> 5, c = lane & 31;
  const int wlane = nblk * (int)WBLK + (2 * g) * 2048 + (wave * 32 + c) * 4;
  const int xlane = (2 * g) * 8192 + (mblk * 64 + c) * 16;

  int ready = 0;
  bool failed = false;
  const unsigned long long t0 = __builtin_amdgcn_s_memrealtime();

  auto wait_for = [&](int s, int kb) {
    while (__builtin_amdgcn_readlane(ready, s * 8) < kb + 2) {
      const unsigned f = __hip_atomic_load(myflag, __ATOMIC_RELAXED, __HIP_MEMORY_SCOPE_SYSTEM);
      int v = (int)(f - fbase);
      if (isx) v = v >= 1 ? 1000 : 0;
      v = min(v, __shfl_xor(v, 1));
      v = min(v, __shfl_xor(v, 2));
      v = min(v, __shfl_xor(v, 4));
      ready = v;
      if (__builtin_amdgcn_readlane(ready, s * 8) >= kb + 2) break;
      if (__builtin_amdgcn_readfirstlane(__hip_atomic_load(tmo, __ATOMIC_RELAXED, __HIP_MEMORY_SCOPE_AGENT)) == call ||
          __builtin_amdgcn_s_memrealtime() - t0 > DEADLINE_TICKS) {
        if (lane == 0) __hip_atomic_store(tmo, call, __ATOMIC_RELAXED, __HIP_MEMORY_SCOPE_AGENT);
        failed = true;
        ready = 1000;
        break;
      }
      __builtin_amdgcn_s_sleep(4);
    }
    asm volatile("" ::: "memory");
  };

  auto step_sk = [&](int st, int& s, int& kb) {
    if (st < 8) { s = rank; kb = st; }
    else { const int r = st - 8; kb = r / 7; s = (rank + 1 + (r - kb * 7)) & 7; }
  };

  auto load_frag = [&](Frag& f, int s, int kb) {
    const int wo = s * (int)WSRC + kb * (int)(32 * WBLK);
    const int xo = s * (int)XSRC + kb * (4 * 8192);
#pragma unroll
    for (int h = 0; h < 2; ++h) {
      const v4u t0v = __builtin_amdgcn_raw_buffer_load_b128(rsX, xlane + h * 8192, xo, 0);
      const v4u t1v = __builtin_amdgcn_raw_buffer_load_b128(rsX, xlane + h * 8192 + 512, xo, 0);
#pragma unroll
      for (int e = 0; e < 4; ++e) { f.a0[h * 4 + e] = (int)t0v[e]; f.a1[h * 4 + e] = (int)t1v[e]; }
#pragma unroll
      for (int q = 0; q < 4; ++q)
        f.b[h * 4 + q] = (int)__builtin_amdgcn_raw_buffer_load_b32(rsW, wlane + h * 2048 + q * 512, wo, 0);
    }
  };

  v16f acc0 = {}, acc1 = {};
  Frag fa, fb;
  {
    int s, kb;
    step_sk(0, s, kb);
    wait_for(s, kb);
    load_frag(fa, s, kb);
  }
  for (int st = 0; st < 64; st += 2) {
    {
      int s, kb;
      step_sk(st + 1, s, kb);
      wait_for(s, kb);
      load_frag(fb, s, kb);
    }
    acc0 = __builtin_amdgcn_mfma_scale_f32_32x32x64_f8f6f4(fa.a0, fa.b, acc0, 0, 1, 0, 127, 0, 127);
    acc1 = __builtin_amdgcn_mfma_scale_f32_32x32x64_f8f6f4(fa.a1, fa.b, acc1, 0, 1, 0, 127, 0, 127);
    if (st + 2 < 64) {
      int s, kb;
      step_sk(st + 2, s, kb);
      wait_for(s, kb);
      load_frag(fa, s, kb);
    }
    acc0 = __builtin_amdgcn_mfma_scale_f32_32x32x64_f8f6f4(fb.a0, fb.b, acc0, 0, 1, 0, 127, 0, 127);
    acc1 = __builtin_amdgcn_mfma_scale_f32_32x32x64_f8f6f4(fb.a1, fb.b, acc1, 0, 1, 0, 127, 0, 127);
  }

  if (__builtin_amdgcn_readfirstlane(__hip_atomic_load(tmo, __ATOMIC_RELAXED, __HIP_MEMORY_SCOPE_AGENT)) == call)
    failed = true;
  const float scale = p.sx[0] * p.sw[0];
  float* const o = p.out + (size_t)(mblk * 64 + 4 * g) * NN + nblk * 128 + wave * 32 + c;
#pragma unroll
  for (int u = 0; u < 2; ++u) {
#pragma unroll
    for (int r = 0; r < 16; ++r) {
      const float y = (u ? acc1[r] : acc0[r]) * scale;
      const float z = fminf(fmaxf(y, -60.0f), 60.0f);
      float v = y / (1.0f + __expf(-z));
      if (failed) v = __builtin_nanf("");
      o[(size_t)(u * 32 + (r & 3) + 8 * (r >> 2)) * NN] = v;
    }
  }
}

extern "C" void kernel_launch(int rank, int nranks, ncclComm_t comm,
                              hipStream_t stream,
                              void* const* d_in, const int* in_sizes, int n_in,
                              void* d_out, int out_size,
                              const int* in_dtypes, int out_dtype,
                              void* const* d_out_all,
                              void* const* d_ws_all, size_t ws_bytes) {
  (void)comm; (void)in_sizes; (void)n_in; (void)out_size; (void)in_dtypes; (void)out_dtype; (void)d_out_all;
  if (nranks != NR || (unsigned)rank >= (unsigned)NR || ws_bytes < WS_NEED) {
    fprintf(stderr, "kernel_launch: unexpected configuration\n");
    return;
  }
  Params p;
  p.x = (const uint8_t*)d_in[0];
  p.w = (const uint8_t*)d_in[1];
  p.sx = (const float*)d_in[2];
  p.sw = (const float*)d_in[3];
  p.out = (float*)d_out;
  for (int i = 0; i < NR; ++i) p.ws[i] = (uint8_t*)d_ws_all[i];
  p.rank = rank;
  hipLaunchKernelGGL(fused_ag_gemm, dim3(NWG), dim3(NTHREADS), 0, stream, p);
  HIP_CHECK(hipGetLastError());
}
```

```cpp
#include <hip/hip_runtime.h>
#include <rccl/rccl.h>
#include <cstdio>
#include <cstdint>

#define HIP_CHECK(expr) do { hipError_t _e = (expr); if (_e != hipSuccess) { \
    fprintf(stderr, "HIP error %s at %s:%d\n", hipGetErrorString(_e), __FILE__, __LINE__); \
    return; } } while (0)

typedef int      v8i  __attribute__((ext_vector_type(8)));
typedef unsigned v4u  __attribute__((ext_vector_type(4)));
typedef float    v16f __attribute__((ext_vector_type(16)));
#define LAS __attribute__((address_space(3)))

namespace {

constexpr int NR = 8;
constexpr int NN = 4096;
constexpr int MS = 512;
constexpr int KS = 512;
constexpr int NWG = 256;
constexpr int NTHREADS = 960;

constexpr size_t WFLAG_OFF = 0;
constexpr size_t XFLAG_OFF = 32768;
constexpr size_t TMO_OFF   = 40960;
constexpr size_t EPOCH_OFF = 49152;
constexpr size_t SLOT_OFF  = 1u << 20;
constexpr size_t WBLK      = 8192;
constexpr size_t WSRC      = 8 * 32 * WBLK;
constexpr size_t XSRC      = 32 * 512 * 16;
constexpr size_t X_OFF     = NR * WSRC;
constexpr size_t SLOT_BYTES = X_OFF + NR * XSRC;
constexpr size_t WS_NEED   = SLOT_OFF + 2 * SLOT_BYTES;

constexpr int RING = 5;
constexpr int SLOTB = 12288;
constexpr int FLY = 2;
constexpr int L_RING = 0;
constexpr int L_PRIV = 2 * RING * SLOTB;
constexpr int L_SLIST = L_PRIV + 8 * 4096;
constexpr int L_CTRL = L_SLIST + 528;
constexpr int C_ANY = 0;
constexpr int C_NSEQ = 8;
constexpr int C_FAIL = 12;
constexpr int C_RAWH = 16;
constexpr int C_PROG = 32;
constexpr int C_FULL = 64;
constexpr int C_DONE = 80;
constexpr int C_SEQ = 112;
constexpr int L_TOTAL = L_CTRL + 176;

constexpr unsigned long long DEADLINE_TICKS = 50000000ull;
constexpr int AUX_SYS = 17;
constexpr int RSRC3 = 0x00020000;
#define POLL_SLEEP 4
#define FORCE_RAW 0
#define PACE 117
#define T_START 90

struct Params {
  const uint8_t* x;
  const uint8_t* w;
  const float* sx;
  const float* sw;
  float* out;
  uint8_t* ws[NR];
  int rank;
};

#define TS(idx) do {} while (0)
#define STR2(x) #x
#define STR(x) STR2(x)
#define WAIT_VM(n) asm volatile("s_waitcnt vmcnt(" STR(n) ")" ::: "memory")

__device__ __forceinline__ v4u transpose4(unsigned a0, unsigned a1, unsigned a2, unsigned a3) {
  const unsigned t0 = __builtin_amdgcn_perm(a1, a0, 0x05010400u);
  const unsigned t1 = __builtin_amdgcn_perm(a1, a0, 0x07030602u);
  const unsigned t2 = __builtin_amdgcn_perm(a3, a2, 0x05010400u);
  const unsigned t3 = __builtin_amdgcn_perm(a3, a2, 0x07030602u);
  v4u r;
  r[0] = __builtin_amdgcn_perm(t2, t0, 0x05040100u);
  r[1] = __builtin_amdgcn_perm(t2, t0, 0x07060302u);
  r[2] = __builtin_amdgcn_perm(t3, t1, 0x05040100u);
  r[3] = __builtin_amdgcn_perm(t3, t1, 0x07060302u);
  return r;
}
__device__ __forceinline__ unsigned bfi(unsigned mask, unsigned a, unsigned b) { return (a & mask) | (b & ~mask); }
__device__ __forceinline__ unsigned long long uni64(unsigned lo, unsigned hi) {
  return ((unsigned long long)(unsigned)__builtin_amdgcn_readfirstlane(hi) << 32) |
         (unsigned)__builtin_amdgcn_readfirstlane(lo);
}

}

__global__ void __launch_bounds__(NTHREADS, 1) fused_ag_gemm(Params p) {
  __shared__ __attribute__((aligned(16))) unsigned char smem_[L_TOTAL];
  LAS unsigned char* const smem = (LAS unsigned char*)smem_;
  LAS unsigned char* const ctrl = smem + L_CTRL;
  const int b = blockIdx.x;
  const int wave = __builtin_amdgcn_readfirstlane(threadIdx.x >> 6);
  const int lane = threadIdx.x & 63;
  const int rank = p.rank;
  uint8_t* const myws = p.ws[rank];

  TS(0);
  unsigned* const ep = (unsigned*)(myws + EPOCH_OFF) + b * 16 + wave;
  const unsigned epv = __hip_atomic_load(ep, __ATOMIC_RELAXED, __HIP_MEMORY_SCOPE_AGENT);
  unsigned* const tmo = (unsigned*)(myws + TMO_OFF);

  if (wave == 12 || wave == 13) {
    const unsigned long long tstart = __builtin_amdgcn_s_memrealtime();
    const int hw = wave - 12;
    const int a = lane & 15, kc = lane >> 4;
    const int dx = b >> 5, ix = b & 31;
    const auto rsW = __builtin_amdgcn_make_buffer_rsrc((void*)p.w, 0, KS * NN, RSRC3);
    const auto rsX = __builtin_amdgcn_make_buffer_rsrc((void*)p.x, 0, 4096 * KS, RSRC3);
    v4u xr[4];
    {
      const int off = (dx * MS + (ix >> 2) * 64 + lane) * KS + (ix & 3) * 128 + hw * 64;
#pragma unroll
      for (int t = 0; t < 4; ++t) xr[t] = __builtin_amdgcn_raw_buffer_load_b128(rsX, off + t * 16, 0, 0);
    }
    unsigned cur[16];
    {
      const int woff = ((b >> 5) * 64 + kc * 16) * NN + (b & 31) * 128 + hw * 64 + a * 4;
#pragma unroll
      for (int t = 0; t < 16; ++t) cur[t] = __builtin_amdgcn_raw_buffer_load_b32(rsW, woff + t * NN, 0, 0);
    }
    __builtin_amdgcn_s_barrier();
    __builtin_amdgcn_s_setprio(3);
    const unsigned call0 = __builtin_amdgcn_readfirstlane(epv);
    const unsigned call = call0 + 1;
    const unsigned par = call0 & 1;
    if (lane == 0) __hip_atomic_store(ep, call, __ATOMIC_RELAXED, __HIP_MEMORY_SCOPE_AGENT);
    const size_t soff = SLOT_OFF + (size_t)par * SLOT_BYTES;
    const unsigned w64 = (unsigned)(ix * 2 + hw);
    auto wait_slot = [&](unsigned ticks) {
      while ((long long)(__builtin_amdgcn_s_memrealtime() - (tstart + ticks)) < 0) __builtin_amdgcn_s_sleep(1);
    };
    uint8_t* const dws = p.ws[dx];
    const bool remote = dx != rank;
    const auto rdX = __builtin_amdgcn_make_buffer_rsrc((void*)(dws + soff + X_OFF + (size_t)rank * XSRC), 0, (int)XSRC, RSRC3);
    const int xoff = (((ix & 3) * 2 + hw) * 8 + (ix >> 2)) * 4096 + lane * 16;
    unsigned* const xflag = (unsigned*)(dws + XFLAG_OFF) + par * 512 + ((ix >> 2) * 8 + rank) * 8 + (ix & 3) * 2 + hw;
    const unsigned xslot = T_START + (w64 * 4u * PACE) / 64u;
    const bool xearly = !remote || w64 < 20u;
    if (xearly) {
      if (remote) wait_slot(xslot);
#pragma unroll
      for (int t = 0; t < 4; ++t) __builtin_amdgcn_raw_buffer_store_b128(xr[t], rdX, xoff + t * 1024, 0, AUX_SYS);
    }

    v4u pk[4];
    unsigned cnt = 0;
    bool raw;
    {
      const unsigned rm = 0x00ff00ffu << (a & 8);
      const unsigned lt = (1u << (lane & 31)) - 1u;
      const unsigned below_lo = lane < 32 ? (rm & lt) : rm;
      const unsigned below_hi = lane < 32 ? 0u : (rm & lt);
      const unsigned posbase = (unsigned)(((kc & 1) << 10) | (((kc >> 1) * 32 + ((4 * a) & 31)) << 4));
      const unsigned lbase = (unsigned)(L_SLIST + (hw * 2 + (a >> 3)) * 128);
      unsigned s = 0;
#pragma unroll
      for (int t = 0; t < 16; t += 4) s = __builtin_amdgcn_sad_u8((cur[t] >> 2) & 0x1f1f1f1fu, 0u, s);
      int tot = 96;
#pragma unroll
      for (int bit = 2; bit < 9; ++bit) tot += __builtin_popcountll(__ballot((s >> bit) & 1u)) << bit;
      int base = (tot - 4291) >> 10;
      base = base < 0 ? 0 : (base > 24 ? 24 : base);
      const unsigned b4 = (unsigned)base * 0x04040404u;
      unsigned ap[16], ex[4];
#pragma unroll
      for (int t = 0; t < 16; ++t) {
        const unsigned dd = (cur[t] | 0x80808080u) - b4;
        const int tt = t & 3;
        const unsigned e2 = tt == 0 ? ((dd >> 5) & 0x03030303u) : tt == 1 ? ((dd >> 3) & 0x0c0c0c0cu)
                          : tt == 2 ? ((dd >> 1) & 0x30303030u) : ((dd << 1) & 0xc0c0c0c0u);
        ex[t >> 2] = tt == 0 ? e2 : (ex[t >> 2] | e2);
        ap[t] = bfi(0x80808080u, cur[t], dd) & 0x9f9f9f9fu;
      }
#pragma unroll
      for (int w = 0; w < 4; ++w) {
        unsigned e = ex[w];
        for (;;) {
          const unsigned long long m = __ballot(e != 0u);
          if (!m) break;
          const unsigned mlo = (unsigned)m, mhi = (unsigned)(m >> 32);
          const unsigned slotn = cnt + 1u + __builtin_popcount(mlo & below_lo) + __builtin_popcount(mhi & below_hi);
          const unsigned bb = (unsigned)__builtin_ctz(e | 0x80000000u) & 30u;
          const unsigned hc = (e >> bb) & 3u;
          const unsigned pos = (unsigned)(4 * w) + posbase + ((bb & 0x18u) << 1) + ((bb >> 1) & 3u);
          const unsigned la = (e != 0u && slotn <= 63u) ? lbase + 2u * slotn : (unsigned)(L_SLIST + 512);
          *(LAS unsigned short*)(smem + la) = (unsigned short)((pos << 2) | hc);
          e ^= hc << bb;
          cnt += __builtin_popcount(mlo & rm) + __builtin_popcount(mhi & rm);
        }
      }
      raw = FORCE_RAW || __ballot(cnt > 63u) != 0ull;
      if (raw) {
#pragma unroll
        for (int q = 0; q < 4; ++q) pk[q] = transpose4(cur[4 * q], cur[4 * q + 1], cur[4 * q + 2], cur[4 * q + 3]);
      } else {
        if ((lane & 55) == 0) *(LAS unsigned short*)(smem + lbase) = (unsigned short)(cnt | ((unsigned)base << 6));
#pragma unroll
        for (int r = 0; r < 4; ++r) {
          const unsigned v = ap[12 + r];
          ap[0 + r] |= (v << 4) & 0x60606060u;
          ap[4 + r] |= (v << 2) & 0x60606060u;
          ap[8 + r] |= ((v >> 1) & 0x40404040u) | ((v << 5) & 0x20202020u);
        }
#pragma unroll
        for (int q = 0; q < 3; ++q) pk[q] = transpose4(ap[4 * q], ap[4 * q + 1], ap[4 * q + 2], ap[4 * q + 3]);
        asm volatile("" ::: "memory");
        pk[3] = *(LAS v4u*)(smem + L_SLIST + hw * 256 + a * 16);
      }
    }
    TS(41);

    const auto rlW = __builtin_amdgcn_make_buffer_rsrc((void*)(myws + soff + (size_t)rank * WSRC), 0, (int)WSRC, RSRC3);
    const auto rdW = __builtin_amdgcn_make_buffer_rsrc((void*)(dws + soff + (size_t)rank * WSRC), 0, (int)WSRC, RSRC3);
    unsigned* const lflags = (unsigned*)(myws + WFLAG_OFF) + par * 4096 + rank * 16;
    unsigned* const dflags = (unsigned*)(dws + WFLAG_OFF) + par * 4096 + rank * 16;
    auto fidx = [&](unsigned it) { return ((it >> 1) & 31u) * 128u + (it >> 6) * 2u + (it & 1u); };
    auto off0 = [&](unsigned it, bool r) { return (int)(it >> 1) * (int)WBLK + kc * (r ? 2048 : 1536) + (int)(it & 1u) * 256 + a * 16; };
    auto off3 = [&](unsigned it, bool r) {
      return r ? off0(it, true) + 1536 : (int)(it >> 1) * (int)WBLK + 6144 + (int)(it & 1u) * 256 + a * 16; };
    unsigned item = (unsigned)b * 2u + (unsigned)hw;
    unsigned fv = call * 2 + (raw ? 1u : 0u);
    {
      const int db = off0(item, raw), db3 = off3(item, raw);
#pragma unroll
      for (int q = 0; q < 3; ++q) __builtin_amdgcn_raw_buffer_store_b128(pk[q], rlW, db + q * 512, 0, AUX_SYS);
      if (raw || (lane < 16 && (unsigned)(lane & 7) * 8u <= cnt))
        __builtin_amdgcn_raw_buffer_store_b128(pk[3], rlW, db3, 0, AUX_SYS);
      WAIT_VM(0);
      if (lane == 0) {
        __hip_atomic_store(lflags + fidx(item), fv, __ATOMIC_RELAXED, __HIP_MEMORY_SCOPE_SYSTEM);
        if (xearly) __hip_atomic_store(xflag, call, __ATOMIC_RELAXED, __HIP_MEMORY_SCOPE_SYSTEM);
      }
      TS(42);
    }
    if (!xearly) {
      wait_slot(xslot);
#pragma unroll
      for (int t = 0; t < 4; ++t) __builtin_amdgcn_raw_buffer_store_b128(xr[t], rdX, xoff + t * 1024, 0, AUX_SYS);
      WAIT_VM(0);
      if (lane == 0) __hip_atomic_store(xflag, call, __ATOMIC_RELAXED, __HIP_MEMORY_SCOPE_SYSTEM);
    }
    if (remote) {
      const unsigned col = (w64 + (unsigned)rank * 8u) & 63u;
      unsigned slot_t = T_START + 4u * PACE + (w64 * (16u * PACE / 5u)) / 64u;
      const unsigned litem = (unsigned)((lane & 7) < dx ? (lane & 7) : (lane & 7) + 1) * 64u + col;
      const unsigned* const lf = lflags + fidx(litem & 511u);
      unsigned lfv = 0;
#pragma unroll 1
      for (int n = 0; n < 8; ++n) {
        const bool r = fv & 1u;
        const int db = off0(item, r), db3 = off3(item, r);
        const unsigned lc = (unsigned)__shfl((int)pk[3][0], lane & 8) & 63u;
        unsigned nitem = 0, nfv = 0;
        v4u n0 = pk[0], n1 = pk[1], n2 = pk[2], n3 = pk[3];
        if (n < 7) {
          nitem = (unsigned)(n < dx ? n : n + 1) * 64u + col;
          while ((int)((nfv = __builtin_amdgcn_readlane(lfv, n)) - 2u * call) < 0) {
            lfv = __hip_atomic_load(lf, __ATOMIC_RELAXED, __HIP_MEMORY_SCOPE_AGENT);
            if ((int)(__builtin_amdgcn_readlane(lfv, n) - 2u * call) >= 0) continue;
            if (__builtin_amdgcn_s_memrealtime() - tstart > DEADLINE_TICKS) {
              if (lane == 0) __hip_atomic_store(tmo, call, __ATOMIC_RELAXED, __HIP_MEMORY_SCOPE_AGENT);
              nfv = 0;
              break;
            }
            __builtin_amdgcn_s_sleep(4);
          }
          asm volatile("" ::: "memory");
          const bool nr = nfv & 1u;
          const int nb0 = off0(nitem, nr), nb3 = off3(nitem, nr);
          n0 = __builtin_amdgcn_raw_buffer_load_b128(rlW, nb0, 0, 0);
          n1 = __builtin_amdgcn_raw_buffer_load_b128(rlW, nb0 + 512, 0, 0);
          n2 = __builtin_amdgcn_raw_buffer_load_b128(rlW, nb0 + 1024, 0, 0);
          n3 = __builtin_amdgcn_raw_buffer_load_b128(rlW, nb3, 0, 0);
        }
        wait_slot(slot_t);
        slot_t += r ? 4u * PACE : 16u * PACE / 5u;
#pragma unroll
        for (int q = 0; q < 3; ++q) __builtin_amdgcn_raw_buffer_store_b128(pk[q], rdW, db + q * 512, 0, AUX_SYS);
        if (r || (lane < 16 && (unsigned)(lane & 7) * 8u <= lc))
          __builtin_amdgcn_raw_buffer_store_b128(pk[3], rdW, db3, 0, AUX_SYS);
        WAIT_VM(0);
        if (lane == 0 && fv) __hip_atomic_store(dflags + fidx(item), fv, __ATOMIC_RELAXED, __HIP_MEMORY_SCOPE_SYSTEM);
        TS(1 + n);
        pk[0] = n0; pk[1] = n1; pk[2] = n2; pk[3] = n3;
        item = nitem;
        fv = nfv;
      }
    }
    TS(40);
    return;
  }

  if (wave == 14 && lane < 7) *(LAS v4u*)(ctrl + lane * 16) = v4u{0u, 0u, 0u, 0u};
  asm volatile("s_waitcnt lgkmcnt(0)" ::: "memory");
  __builtin_amdgcn_s_barrier();

  const unsigned call0 = __builtin_amdgcn_readfirstlane(epv);
  const unsigned call = call0 + 1;
  const unsigned par = call0 & 1;
  if (lane == 0) __hip_atomic_store(ep, call, __ATOMIC_RELAXED, __HIP_MEMORY_SCOPE_AGENT);
  const int nblk = (b & 7) * 4 + ((b >> 3) & 3);
  const int mblk = b >> 5;
  volatile LAS unsigned* const vfail = (volatile LAS unsigned*)(ctrl + C_FAIL);

  if (wave == 14) {
    const unsigned long long* const wfl = (const unsigned long long*)(myws + WFLAG_OFF) + par * 2048 + nblk * 64 + lane;
    const unsigned* const xfl = (const unsigned*)(myws + XFLAG_OFF) + par * 512 + mblk * 64 + lane;
    unsigned long long xok = 0, seen = 0, raw0 = 0, raw1 = 0;
    int nseq = 0;
    const unsigned long long t0 = __builtin_amdgcn_s_memrealtime();
    for (;;) {
      const unsigned long long wf = __hip_atomic_load(wfl, __ATOMIC_RELAXED, __HIP_MEMORY_SCOPE_SYSTEM);
      if (~xok) {
        const unsigned xf = __hip_atomic_load(xfl, __ATOMIC_RELAXED, __HIP_MEMORY_SCOPE_SYSTEM);
        xok = __ballot((int)(xf - call) >= 0);
      }
      const unsigned w0 = (unsigned)wf, w1 = (unsigned)(wf >> 32);
      unsigned long long rdy = __ballot((int)(w0 - 2u * call) >= 0 && (int)(w1 - 2u * call) >= 0) & xok;
      const unsigned long long fresh = rdy & ~seen;
      if (fresh) {
        seen = rdy;
        const int pos = nseq + __builtin_popcountll(fresh & ((1ull << lane) - 1ull));
        if ((fresh >> lane) & 1ull) *(volatile LAS unsigned char*)(ctrl + C_SEQ + pos) = (unsigned char)lane;
        const unsigned long long m0 = __ballot(w0 & 1u), m1 = __ballot(w1 & 1u);
        unsigned long long rw = (m0 | m1) & fresh;
        if (rw) {
          while (rw) {
            const int st = __builtin_ctzll(rw);
            rw &= rw - 1;
            const int ps = nseq + __builtin_popcountll(fresh & ((1ull << st) - 1ull));
            raw0 |= ((m0 >> st) & 1ull) << ps;
            raw1 |= ((m1 >> st) & 1ull) << ps;
          }
          if (lane == 0) {
            *(volatile LAS unsigned long long*)(ctrl + C_RAWH) = raw0;
            *(volatile LAS unsigned long long*)(ctrl + C_RAWH + 8) = raw1;
            *(volatile LAS unsigned long long*)(ctrl + C_ANY) = raw0 | raw1;
          }
        }
        nseq += __builtin_popcountll(fresh);
        if (lane == 0) *(volatile LAS unsigned*)(ctrl + C_NSEQ) = (unsigned)nseq;
        if (!~rdy) break;
        continue;
      }
      const unsigned long long dt = __builtin_amdgcn_s_memrealtime() - t0;
      if (dt > 20000) {
        if (__builtin_amdgcn_readfirstlane(__hip_atomic_load(tmo, __ATOMIC_RELAXED, __HIP_MEMORY_SCOPE_AGENT)) == call ||
            dt > DEADLINE_TICKS) {
          if (lane == 0) {
            __hip_atomic_store(tmo, call, __ATOMIC_RELAXED, __HIP_MEMORY_SCOPE_AGENT);
            *vfail = 1;
          }
          break;
        }
        __builtin_amdgcn_s_sleep(127);
        __builtin_amdgcn_s_sleep(127);
      } else {
        __builtin_amdgcn_s_sleep(POLL_SLEEP);
      }
    }
    return;
  }

  uint8_t* const slot = myws + SLOT_OFF + (size_t)par * SLOT_BYTES;

  if (wave >= 8) {
    const int L = (wave - 8) >> 1, half = wave & 1;
    const unsigned ring_addr = (unsigned)(size_t)(smem + L_RING) + (unsigned)L * (RING * SLOTB);
    const unsigned vo = (unsigned)lane * 16u;
    const uint8_t* const xsrc = slot + X_OFF + (size_t)mblk * 4096;
    const uint8_t* const wsrc = slot + (size_t)nblk * WBLK;
    volatile LAS unsigned* const vfull = (volatile LAS unsigned*)(ctrl + C_FULL + (wave - 8) * 4);
    unsigned long long posraw = 0;
    int seqv = 0;
    int j = half, avail = 0, nis = 0, full = 0;
    unsigned dst = ring_addr + (unsigned)half * SLOTB;
    WAIT_VM(0);
    while (full < 16) {
      const v4u ct = *(volatile LAS v4u*)(ctrl + C_ANY);
      const v4u pr = *(volatile LAS v4u*)(ctrl + C_PROG + L * 16);
      const int sv = *(volatile LAS unsigned char*)(ctrl + C_SEQ + lane);
      const bool go = j < avail;
      if (go) {
        const int pos = 2 * j + L;
        const int st = __builtin_amdgcn_readlane(seqv, pos);
        const uint8_t* const xa = xsrc + (size_t)st * 32768;
        const uint8_t* const wa = wsrc + (size_t)st * (32 * WBLK);
        const uint8_t* const wb = wa + 4096;
        unsigned keep;
        asm volatile(
            "s_nop 4\n\t"
            "s_mov_b32 %0, m0\n\t"
            "s_mov_b32 m0, %5\n\t"
            "s_nop 0\n\t"
            "global_load_lds_dwordx4 %1, %2\n\t"
            "global_load_lds_dwordx4 %1, %2 offset:1024\n\t"
            "global_load_lds_dwordx4 %1, %2 offset:2048\n\t"
            "global_load_lds_dwordx4 %1, %2 offset:3072\n\t"
            "s_add_u32 m0, m0, 0x1000\n\t"
            "s_nop 0\n\t"
            "global_load_lds_dwordx4 %1, %3\n\t"
            "global_load_lds_dwordx4 %1, %3 offset:1024\n\t"
            "global_load_lds_dwordx4 %1, %3 offset:2048\n\t"
            "global_load_lds_dwordx4 %1, %3 offset:3072\n\t"
            "s_add_u32 m0, m0, 0x1000\n\t"
            "s_nop 0\n\t"
            "global_load_lds_dwordx4 %1, %4\n\t"
            "global_load_lds_dwordx4 %1, %4 offset:1024\n\t"
            "global_load_lds_dwordx4 %1, %4 offset:2048\n\t"
            "s_mov_b32 m0, %0"
            : "=&s"(keep)
            : "v"(vo), "s"(xa), "s"(wa), "s"(wb), "s"(dst)
            : "memory", "scc");
        if ((posraw >> pos) & 1ull) {
          asm volatile(
              "s_mov_b32 %0, m0\n\t"
              "s_mov_b32 m0, %3\n\t"
              "s_nop 0\n\t"
              "global_load_lds_dwordx4 %1, %2 offset:3072\n\t"
              "s_mov_b32 m0, %0"
              : "=&s"(keep)
              : "v"(vo), "s"(wb), "s"(dst + 8192u)
              : "memory");
        }
        j += 2;
        ++nis;
        dst += 2 * SLOTB;
        if (dst >= ring_addr + RING * SLOTB) dst -= RING * SLOTB;
        if (nis - full == FLY) {
          WAIT_VM(11);
          ++full;
          if (lane == 0) *vfull = (unsigned)full;
        }
      }
      seqv = sv;
      const int lim = (int)min(min(pr[0], pr[1]), min(pr[2], pr[3])) + RING;
      avail = __builtin_amdgcn_readfirstlane(min(lim, ((int)ct[2] + 1 - L) >> 1));
      posraw = uni64(ct[0], ct[1]);
      if (go || j < avail) continue;
      if (nis > full) {
        WAIT_VM(0);
        full = nis;
        if (lane == 0) *vfull = (unsigned)full;
        continue;
      }
      if (__builtin_amdgcn_readfirstlane((int)ct[3])) break;
      __builtin_amdgcn_s_sleep(1);
    }
    return;
  }

  const int wc = wave & 3, pp = wave >> 2;
  const int g = lane >> 5, c = lane & 31;
  LAS unsigned char* const priv = smem + L_PRIV + wave * 4096;
  *(LAS v4u*)(priv + lane * 16) = v4u{0u, 0u, 0u, 0u};
  *(LAS v4u*)(priv + 1024 + lane * 16) = v4u{0u, 0u, 0u, 0u};
  const float scale = p.sx[0] * p.sw[0];
  const LAS unsigned char* const ring = smem + L_RING + pp * (RING * SLOTB);
  const int aoff = (2 * g) * 1024 + c * 16;
  const int roff = 4096 + (8 * g) * 512 + wc * 128 + c * 4;
  const int poff = 4096 + (6 * g) * 512 + wc * 128 + c * 4;
  const int loff = 4096 + 12 * 512 + wc * 128 + lane * 2;
  volatile LAS unsigned* const vprog = (volatile LAS unsigned*)(ctrl + C_PROG + wave * 4);

  v16f acc0 = {}, acc1 = {};
  int nfull = 0;
  unsigned long long posraw = 0;
  int so = 0;
  for (int j = 0; j < 32; ++j) {
    while (nfull <= j) {
      const unsigned long long f2 = *(volatile LAS unsigned long long*)(ctrl + C_FULL + pp * 8);
      const unsigned f = min(2u * (unsigned)f2, 2u * (unsigned)(f2 >> 32) + 1u);
      const unsigned long long r = *(volatile LAS unsigned long long*)(ctrl + C_RAWH + (wc >> 1) * 8);
      const unsigned fl = *vfail;
      nfull = __builtin_amdgcn_readfirstlane((int)(fl ? 32u : f));
      posraw = uni64((unsigned)r, (unsigned)(r >> 32)) >> pp;
      if (nfull > j) break;
      __builtin_amdgcn_s_sleep(1);
    }
    asm volatile("" ::: "memory");
    const LAS unsigned char* const sl = ring + so;
    v8i a0, a1, bf;
    if ((posraw >> (2 * j)) & 1ull) {
#pragma unroll
      for (int q = 0; q < 8; ++q) bf[q] = *(const LAS int*)(sl + roff + q * 512);
    } else {
      const unsigned ent = *(const LAS unsigned short*)(sl + loff);
      unsigned pa[6];
#pragma unroll
      for (int q = 0; q < 6; ++q) pa[q] = *(const LAS unsigned*)(sl + poff + q * 512);
      const unsigned hdr = __builtin_amdgcn_readfirstlane(ent);
      const unsigned b4 = ((hdr >> 6) & 31u) * 0x04040404u;
      const unsigned ca = (unsigned)(lane - 1) < (hdr & 63u) ? ((ent >> 2) & 2047u) : 2048u + (unsigned)lane * 4u;
      priv[ca] = (unsigned char)((ent & 3u) << 5);
      const v4u c0 = *(const LAS v4u*)(priv + lane * 16);
      const v4u c1 = *(const LAS v4u*)(priv + 1024 + lane * 16);
      priv[ca] = 0;
#pragma unroll
      for (int hh = 0; hh < 2; ++hh) {
        const v4u cc = hh ? c1 : c0;
        const unsigned p0 = pa[3 * hh], p1 = pa[3 * hh + 1], p2 = pa[3 * hh + 2];
        bf[4 * hh + 0] = (int)bfi(0x80808080u, p0, ((p0 & 0x1f1f1f1fu) | cc[0]) + b4);
        bf[4 * hh + 1] = (int)bfi(0x80808080u, p1, ((p1 & 0x1f1f1f1fu) | cc[1]) + b4);
        bf[4 * hh + 2] = (int)bfi(0x80808080u, p2, ((p2 & 0x1f1f1f1fu) | cc[2]) + b4);
        const unsigned u = ((p1 >> 2) & 0x18181818u) | ((p0 >> 4) & 0x06060606u) | ((p2 >> 5) & 0x01010101u) | cc[3];
        bf[4 * hh + 3] = (int)bfi(0x80808080u, p2 << 1, u + b4);
      }
    }
#pragma unroll
    for (int hh = 0; hh < 2; ++hh) {
      const v4u t0v = *(const LAS v4u*)(sl + aoff + hh * 1024);
      const v4u t1v = *(const LAS v4u*)(sl + aoff + hh * 1024 + 512);
#pragma unroll
      for (int e = 0; e < 4; ++e) { a0[hh * 4 + e] = (int)t0v[e]; a1[hh * 4 + e] = (int)t1v[e]; }
    }
    asm volatile("" ::: "memory");
    if (lane == 0) *vprog = (unsigned)(j + 1);
    acc0 = __builtin_amdgcn_mfma_scale_f32_32x32x64_f8f6f4(a0, bf, acc0, 0, 1, 0, 127, 0, 127);
    acc1 = __builtin_amdgcn_mfma_scale_f32_32x32x64_f8f6f4(a1, bf, acc1, 0, 1, 0, 127, 0, 127);
    so = so + SLOTB == RING * SLOTB ? 0 : so + SLOTB;
  }

  TS(70);
  {
    const v16f give = pp ? acc0 : acc1;
#pragma unroll
    for (int q = 0; q < 4; ++q) {
      v4u v;
#pragma unroll
      for (int e = 0; e < 4; ++e) v[e] = __float_as_uint(give[4 * q + e]);
      *(LAS v4u*)(priv + q * 1024 + lane * 16) = v;
    }
    asm volatile("" ::: "memory");
    if (lane == 0) *(volatile LAS unsigned*)(ctrl + C_DONE + wave * 4) = 1u;
    while (!__builtin_amdgcn_readfirstlane((int)*(volatile LAS unsigned*)(ctrl + C_DONE + (wave ^ 4) * 4)))
      __builtin_amdgcn_s_sleep(1);
    asm volatile("" ::: "memory");
  }
  const bool failed = __builtin_amdgcn_readfirstlane((int)*vfail) != 0;
  const LAS unsigned char* const other = smem + L_PRIV + (wave ^ 4) * 4096;
  float* const o = p.out + (size_t)(mblk * 64 + pp * 32 + 4 * g) * NN + nblk * 128 + wc * 32 + c;
#pragma unroll
  for (int q = 0; q < 4; ++q) {
    const v4u ov = *(const LAS v4u*)(other + q * 1024 + lane * 16);
#pragma unroll
    for (int e = 0; e < 4; ++e) {
      const int r = 4 * q + e;
      const float y = ((pp ? acc1[r] : acc0[r]) + __uint_as_float(ov[e])) * scale;
      const float z = fminf(fmaxf(y, -60.0f), 60.0f);
      float v = y * __builtin_amdgcn_rcpf(1.0f + __expf(-z));
      if (failed) v = __builtin_nanf("");
      o[(size_t)(e + 8 * q) * NN] = v;
    }
  }
}

extern "C" void kernel_launch(int rank, int nranks, ncclComm_t comm,
                              hipStream_t stream,
                              void* const* d_in, const int* in_sizes, int n_in,
                              void* d_out, int out_size,
                              const int* in_dtypes, int out_dtype,
                              void* const* d_out_all,
                              void* const* d_ws_all, size_t ws_bytes) {
  (void)comm; (void)in_sizes; (void)n_in; (void)out_size; (void)in_dtypes; (void)out_dtype; (void)d_out_all;
  if (nranks != NR || (unsigned)rank >= (unsigned)NR || ws_bytes < WS_NEED) {
    fprintf(stderr, "kernel_launch: unexpected configuration\n");
    return;
  }
  Params p;
  p.x = (const uint8_t*)d_in[0];
  p.w = (const uint8_t*)d_in[1];
  p.sx = (const float*)d_in[2];
  p.sw = (const float*)d_in[3];
  p.out = (float*)d_out;
  for (int i = 0; i < NR; ++i) p.ws[i] = (uint8_t*)d_ws_all[i];
  p.rank = rank;
  hipLaunchKernelGGL(fused_ag_gemm, dim3(NWG), dim3(NTHREADS), 0, stream, p);
  HIP_CHECK(hipGetLastError());
}
```

```cpp
#include <hip/hip_runtime.h>
#include <rccl/rccl.h>
#include <cstdio>
#include <cstdint>

#define HIP_CHECK(expr) do { hipError_t _e = (expr); if (_e != hipSuccess) { \
    fprintf(stderr, "HIP error %s at %s:%d\n", hipGetErrorString(_e), __FILE__, __LINE__); \
    return; } } while (0)

typedef int      v8i  __attribute__((ext_vector_type(8)));
typedef unsigned v4u  __attribute__((ext_vector_type(4)));
typedef float    v16f __attribute__((ext_vector_type(16)));
#define LAS __attribute__((address_space(3)))

namespace {

constexpr int NR = 8;
constexpr int NN = 4096;
constexpr int MS = 512;
constexpr int KS = 512;
constexpr int NWG = 256;
constexpr int NTHREADS = 960;

constexpr size_t WFLAG_OFF = 0;
constexpr size_t XFLAG_OFF = 32768;
constexpr size_t TMO_OFF   = 40960;
constexpr size_t EPOCH_OFF = 49152;
constexpr size_t SLOT_OFF  = 1u << 20;
constexpr size_t WBLK      = 8192;
constexpr size_t WSRC      = 8 * 32 * WBLK;
constexpr size_t XSRC      = 32 * 512 * 16;
constexpr size_t X_OFF     = NR * WSRC;
constexpr size_t SLOT_BYTES = X_OFF + NR * XSRC;
constexpr size_t WS_NEED   = SLOT_OFF + 2 * SLOT_BYTES;

constexpr int RING = 5;
constexpr int SLOTB = 12288;
constexpr int FLY = 2;
constexpr int L_RING = 0;
constexpr int L_PRIV = 2 * RING * SLOTB;
constexpr int L_SLIST = L_PRIV + 8 * 4096;
constexpr int L_CTRL = L_SLIST + 528;
constexpr int C_ANY = 0;
constexpr int C_NSEQ = 8;
constexpr int C_FAIL = 12;
constexpr int C_RAWH = 16;
constexpr int C_PROG = 32;
constexpr int C_FULL = 64;
constexpr int C_DONE = 80;
constexpr int C_ANYRAW = 176;
constexpr int C_SEQ = 112;
constexpr int L_TOTAL = L_CTRL + 192;

constexpr unsigned long long DEADLINE_TICKS = 50000000ull;
constexpr int AUX_SYS = 17;
constexpr int RSRC3 = 0x00020000;
#define POLL_SLEEP 4
#define FORCE_RAW 0
#define PACE 115
#define SPLIT 1
#define SPLITN 8
#define CONS_SLEEP 1
#define LOAD_SLEEP 1
#define HPRIO 2
#define WIDE_OUT 1
#define TOUCH_OUT 1
#define T_START 90

struct Params {
  const uint8_t* x;
  const uint8_t* w;
  const float* sx;
  const float* sw;
  float* out;
  uint8_t* ws[NR];
  int rank;
};

#define TS(idx) do {} while (0)
#define DMA_HINT ""
#define STR2(x) #x
#define STR(x) STR2(x)
#define WAIT_VM(n) asm volatile("s_waitcnt vmcnt(" STR(n) ")" ::: "memory")

__device__ __forceinline__ v4u transpose4(unsigned a0, unsigned a1, unsigned a2, unsigned a3) {
  const unsigned t0 = __builtin_amdgcn_perm(a1, a0, 0x05010400u);
  const unsigned t1 = __builtin_amdgcn_perm(a1, a0, 0x07030602u);
  const unsigned t2 = __builtin_amdgcn_perm(a3, a2, 0x05010400u);
  const unsigned t3 = __builtin_amdgcn_perm(a3, a2, 0x07030602u);
  v4u r;
  r[0] = __builtin_amdgcn_perm(t2, t0, 0x05040100u);
  r[1] = __builtin_amdgcn_perm(t2, t0, 0x07060302u);
  r[2] = __builtin_amdgcn_perm(t3, t1, 0x05040100u);
  r[3] = __builtin_amdgcn_perm(t3, t1, 0x07060302u);
  return r;
}
__device__ __forceinline__ unsigned bfi(unsigned mask, unsigned a, unsigned b) { return (a & mask) | (b & ~mask); }
__device__ __forceinline__ unsigned long long uni64(unsigned lo, unsigned hi) {
  return ((unsigned long long)(unsigned)__builtin_amdgcn_readfirstlane(hi) << 32) |
         (unsigned)__builtin_amdgcn_readfirstlane(lo);
}

}

__global__ void __launch_bounds__(NTHREADS, 1) fused_ag_gemm(Params p) {
  __shared__ __attribute__((aligned(16))) unsigned char smem_[L_TOTAL];
  LAS unsigned char* const smem = (LAS unsigned char*)smem_;
  LAS unsigned char* const ctrl = smem + L_CTRL;
  const int b = blockIdx.x;
  const int wave = __builtin_amdgcn_readfirstlane(threadIdx.x >> 6);
  const int lane = threadIdx.x & 63;
  const int rank = p.rank;
  uint8_t* const myws = p.ws[rank];

  TS(0);
  unsigned* const ep = (unsigned*)(myws + EPOCH_OFF) + b * 16 + wave;
  const unsigned epv = __hip_atomic_load(ep, __ATOMIC_RELAXED, __HIP_MEMORY_SCOPE_AGENT);
  unsigned* const tmo = (unsigned*)(myws + TMO_OFF);

  if (wave == 12 || wave == 13) {
    const unsigned long long tstart = __builtin_amdgcn_s_memrealtime();
    const int hw = wave - 12;
    const int a = lane & 15, kc = lane >> 4;
    const int dx = b >> 5, ix = b & 31;
    const auto rsW = __builtin_amdgcn_make_buffer_rsrc((void*)p.w, 0, KS * NN, RSRC3);
    const auto rsX = __builtin_amdgcn_make_buffer_rsrc((void*)p.x, 0, 4096 * KS, RSRC3);
    v4u xr[4];
    {
      const int off = (dx * MS + (ix >> 2) * 64 + lane) * KS + (ix & 3) * 128 + hw * 64;
#pragma unroll
      for (int t = 0; t < 4; ++t) xr[t] = __builtin_amdgcn_raw_buffer_load_b128(rsX, off + t * 16, 0, 0);
    }
    unsigned cur[16];
    {
      const int woff = ((b >> 5) * 64 + kc * 16) * NN + (b & 31) * 128 + hw * 64 + a * 4;
#pragma unroll
      for (int t = 0; t < 16; ++t) cur[t] = __builtin_amdgcn_raw_buffer_load_b32(rsW, woff + t * NN, 0, 0);
    }
    __builtin_amdgcn_s_barrier();
    __builtin_amdgcn_s_setprio(3);
    const unsigned call0 = __builtin_amdgcn_readfirstlane(epv);
    const unsigned call = call0 + 1;
    const unsigned par = call0 & 1;
    if (lane == 0) __hip_atomic_store(ep, call, __ATOMIC_RELAXED, __HIP_MEMORY_SCOPE_AGENT);
    const size_t soff = SLOT_OFF + (size_t)par * SLOT_BYTES;
    const unsigned w64 = (unsigned)(ix * 2 + hw);
    auto wait_slot = [&](unsigned ticks) {
      for (;;) {
        const long long left = (long long)(tstart + ticks - __builtin_amdgcn_s_memrealtime());
        if (left <= 0) break;
        if (left > 40) __builtin_amdgcn_s_sleep(10); else __builtin_amdgcn_s_sleep(1);
      }
    };
    uint8_t* const dws = p.ws[dx];
    const bool remote = dx != rank;
    const auto rdX = __builtin_amdgcn_make_buffer_rsrc((void*)(dws + soff + X_OFF + (size_t)rank * XSRC), 0, (int)XSRC, RSRC3);
    const int xoff = (((ix & 3) * 2 + hw) * 8 + (ix >> 2)) * 4096 + lane * 16;
    unsigned* const xflag = (unsigned*)(dws + XFLAG_OFF) + par * 512 + ((ix >> 2) * 8 + rank) * 8 + (ix & 3) * 2 + hw;
    const unsigned xslot = T_START + (w64 * 4u * PACE) / 64u;
    const bool xearly = !remote || w64 < 20u;
    if (xearly) {
      if (remote) wait_slot(xslot);
#pragma unroll
      for (int t = 0; t < 4; ++t) __builtin_amdgcn_raw_buffer_store_b128(xr[t], rdX, xoff + t * 1024, 0, AUX_SYS);
    }

    v4u pk[4];
    unsigned cnt = 0;
    bool raw;
    {
      const unsigned rm = 0x00ff00ffu << (a & 8);
      const unsigned lt = (1u << (lane & 31)) - 1u;
      const unsigned below_lo = lane < 32 ? (rm & lt) : rm;
      const unsigned below_hi = lane < 32 ? 0u : (rm & lt);
      const unsigned posbase = (unsigned)(((kc & 1) << 10) | (((kc >> 1) * 32 + ((4 * a) & 31)) << 4));
      const unsigned lbase = (unsigned)(L_SLIST + (hw * 2 + (a >> 3)) * 128);
      unsigned s = 0;
#pragma unroll
      for (int t = 0; t < 16; t += 4) s = __builtin_amdgcn_sad_u8((cur[t] >> 2) & 0x1f1f1f1fu, 0u, s);
      int tot = 96;
#pragma unroll
      for (int bit = 2; bit < 9; ++bit) tot += __builtin_popcountll(__ballot((s >> bit) & 1u)) << bit;
      int base = (tot - 4291) >> 10;
      base = base < 0 ? 0 : (base > 24 ? 24 : base);
      const unsigned b4 = (unsigned)base * 0x04040404u;
      unsigned ap[16], ex[4];
#pragma unroll
      for (int t = 0; t < 16; ++t) {
        const unsigned dd = (cur[t] | 0x80808080u) - b4;
        const int tt = t & 3;
        const unsigned e2 = tt == 0 ? ((dd >> 5) & 0x03030303u) : tt == 1 ? ((dd >> 3) & 0x0c0c0c0cu)
                          : tt == 2 ? ((dd >> 1) & 0x30303030u) : ((dd << 1) & 0xc0c0c0c0u);
        ex[t >> 2] = tt == 0 ? e2 : (ex[t >> 2] | e2);
        ap[t] = bfi(0x80808080u, cur[t], dd) & 0x9f9f9f9fu;
      }
#pragma unroll
      for (int w = 0; w < 4; ++w) {
        unsigned e = ex[w];
        for (;;) {
          const unsigned long long m = __ballot(e != 0u);
          if (!m) break;
          const unsigned mlo = (unsigned)m, mhi = (unsigned)(m >> 32);
          const unsigned slotn = cnt + 1u + __builtin_popcount(mlo & below_lo) + __builtin_popcount(mhi & below_hi);
          const unsigned bb = (unsigned)__builtin_ctz(e | 0x80000000u) & 30u;
          const unsigned hc = (e >> bb) & 3u;
          const unsigned pos = (unsigned)(4 * w) + posbase + ((bb & 0x18u) << 1) + ((bb >> 1) & 3u);
          const unsigned la = (e != 0u && slotn <= 63u) ? lbase + 2u * slotn : (unsigned)(L_SLIST + 512);
          *(LAS unsigned short*)(smem + la) = (unsigned short)((pos << 2) | hc);
          e ^= hc << bb;
          cnt += __builtin_popcount(mlo & rm) + __builtin_popcount(mhi & rm);
        }
      }
      raw = FORCE_RAW || __ballot(cnt > 63u) != 0ull;
      if (raw) {
#pragma unroll
        for (int q = 0; q < 4; ++q) pk[q] = transpose4(cur[4 * q], cur[4 * q + 1], cur[4 * q + 2], cur[4 * q + 3]);
      } else {
        if ((lane & 55) == 0) *(LAS unsigned short*)(smem + lbase) = (unsigned short)(cnt | ((unsigned)base << 6));
#pragma unroll
        for (int r = 0; r < 4; ++r) {
          const unsigned v = ap[12 + r];
          ap[0 + r] |= (v << 4) & 0x60606060u;
          ap[4 + r] |= (v << 2) & 0x60606060u;
          ap[8 + r] |= ((v >> 1) & 0x40404040u) | ((v << 5) & 0x20202020u);
        }
#pragma unroll
        for (int q = 0; q < 3; ++q) pk[q] = transpose4(ap[4 * q], ap[4 * q + 1], ap[4 * q + 2], ap[4 * q + 3]);
        asm volatile("" ::: "memory");
        pk[3] = *(LAS v4u*)(smem + L_SLIST + hw * 256 + a * 16);
      }
    }
    TS(41);

    const auto rlW = __builtin_amdgcn_make_buffer_rsrc((void*)(myws + soff + (size_t)rank * WSRC), 0, (int)WSRC, RSRC3);
    const auto rdW = __builtin_amdgcn_make_buffer_rsrc((void*)(dws + soff + (size_t)rank * WSRC), 0, (int)WSRC, RSRC3);
    unsigned* const lflags = (unsigned*)(myws + WFLAG_OFF) + par * 4096 + rank * 16;
    unsigned* const dflags = (unsigned*)(dws + WFLAG_OFF) + par * 4096 + rank * 16;
    auto fidx = [&](unsigned it) { return ((it >> 1) & 31u) * 128u + (it >> 6) * 2u + (it & 1u); };
    auto off0 = [&](unsigned it, bool r) { return (int)(it >> 1) * (int)WBLK + kc * (r ? 2048 : 1536) + (int)(it & 1u) * 256 + a * 16; };
    auto off3 = [&](unsigned it, bool r) {
      return r ? off0(it, true) + 1536 : (int)(it >> 1) * (int)WBLK + 6144 + (int)(it & 1u) * 256 + a * 16; };
    unsigned item = (unsigned)b * 2u + (unsigned)hw;
    unsigned fv = call * 2 + (raw ? 1u : 0u);
    {
      const int db = off0(item, raw), db3 = off3(item, raw);
#pragma unroll
      for (int q = 0; q < 3; ++q) __builtin_amdgcn_raw_buffer_store_b128(pk[q], rlW, db + q * 512, 0, AUX_SYS);
      if (raw || (lane < 16 && (unsigned)(lane & 7) * 8u <= cnt))
        __builtin_amdgcn_raw_buffer_store_b128(pk[3], rlW, db3, 0, AUX_SYS);
      WAIT_VM(0);
      if (lane == 0) {
        __hip_atomic_store(lflags + fidx(item), fv, __ATOMIC_RELAXED, __HIP_MEMORY_SCOPE_SYSTEM);
        if (xearly) __hip_atomic_store(xflag, call, __ATOMIC_RELAXED, __HIP_MEMORY_SCOPE_SYSTEM);
      }
      TS(42);
    }
    if (!xearly) {
      wait_slot(xslot);
#pragma unroll
      for (int t = 0; t < 4; ++t) __builtin_amdgcn_raw_buffer_store_b128(xr[t], rdX, xoff + t * 1024, 0, AUX_SYS);
      WAIT_VM(0);
      if (lane == 0) __hip_atomic_store(xflag, call, __ATOMIC_RELAXED, __HIP_MEMORY_SCOPE_SYSTEM);
    }
    if (remote) {
      const unsigned col = (w64 + (unsigned)rank * 8u) & 63u;
      unsigned slot_t = T_START + 4u * PACE + (w64 * (16u * PACE / 5u)) / 64u;
      const unsigned litem = (unsigned)((lane & 7) < dx ? (lane & 7) : (lane & 7) + 1) * 64u + col;
      const unsigned* const lf = lflags + fidx(litem & 511u);
      unsigned lfv = 0;
#pragma unroll 1
      for (int n = 0; n < 8; ++n) {
        const bool r = fv & 1u;
        const int db = off0(item, r), db3 = off3(item, r);
        const unsigned lc = (unsigned)__shfl((int)pk[3][0], lane & 8) & 63u;
        unsigned nitem = 0, nfv = 0;
        v4u n0 = pk[0], n1 = pk[1], n2 = pk[2], n3 = pk[3];
        if (n < 7) {
          nitem = (unsigned)(n < dx ? n : n + 1) * 64u + col;
          while ((int)((nfv = __builtin_amdgcn_readlane(lfv, n)) - 2u * call) < 0) {
            lfv = __hip_atomic_load(lf, __ATOMIC_RELAXED, __HIP_MEMORY_SCOPE_AGENT);
            if ((int)(__builtin_amdgcn_readlane(lfv, n) - 2u * call) >= 0) continue;
            if (__builtin_amdgcn_s_memrealtime() - tstart > DEADLINE_TICKS) {
              if (lane == 0) __hip_atomic_store(tmo, call, __ATOMIC_RELAXED, __HIP_MEMORY_SCOPE_AGENT);
              nfv = 0;
              break;
            }
            __builtin_amdgcn_s_sleep(4);
          }
          asm volatile("" ::: "memory");
          const bool nr = nfv & 1u;
          const int nb0 = off0(nitem, nr), nb3 = off3(nitem, nr);
          n0 = __builtin_amdgcn_raw_buffer_load_b128(rlW, nb0, 0, 0);
          n1 = __builtin_amdgcn_raw_buffer_load_b128(rlW, nb0 + 512, 0, 0);
          n2 = __builtin_amdgcn_raw_buffer_load_b128(rlW, nb0 + 1024, 0, 0);
          n3 = __builtin_amdgcn_raw_buffer_load_b128(rlW, nb3, 0, 0);
        }
        wait_slot(slot_t);
        slot_t += r ? 4u * PACE : 16u * PACE / 5u;
        __builtin_amdgcn_raw_buffer_store_b128(pk[0], rdW, db, 0, AUX_SYS);
        __builtin_amdgcn_raw_buffer_store_b128(pk[1], rdW, db + 512, 0, AUX_SYS);
        if (n < SPLITN) WAIT_VM(1);
        __builtin_amdgcn_raw_buffer_store_b128(pk[2], rdW, db + 1024, 0, AUX_SYS);
        if (r && n < SPLITN) WAIT_VM(1);
        if (r || (lane < 16 && (unsigned)(lane & 7) * 8u <= lc))
          __builtin_amdgcn_raw_buffer_store_b128(pk[3], rdW, db3, 0, AUX_SYS);
        WAIT_VM(0);
        if (lane == 0 && fv) __hip_atomic_store(dflags + fidx(item), fv, __ATOMIC_RELAXED, __HIP_MEMORY_SCOPE_SYSTEM);
        TS(1 + n);
        pk[0] = n0; pk[1] = n1; pk[2] = n2; pk[3] = n3;
        item = nitem;
        fv = nfv;
      }
    }
    TS(40);
    return;
  }

  if (wave == 14 && lane < 12) *(LAS v4u*)(ctrl + lane * 16) = v4u{0u, 0u, 0u, 0u};
  asm volatile("s_waitcnt lgkmcnt(0)" ::: "memory");
  __builtin_amdgcn_s_barrier();

  const unsigned call0 = __builtin_amdgcn_readfirstlane(epv);
  const unsigned call = call0 + 1;
  const unsigned par = call0 & 1;
  if (lane == 0) __hip_atomic_store(ep, call, __ATOMIC_RELAXED, __HIP_MEMORY_SCOPE_AGENT);
  const int nblk = (b & 7) * 4 + ((b >> 3) & 3);
  const int mblk = b >> 5;
  volatile LAS unsigned* const vfail = (volatile LAS unsigned*)(ctrl + C_FAIL);

  if (wave == 14) {
    __builtin_amdgcn_s_setprio(HPRIO);
    const unsigned long long* const wfl = (const unsigned long long*)(myws + WFLAG_OFF) + par * 2048 + nblk * 64 + lane;
    const unsigned* const xfl = (const unsigned*)(myws + XFLAG_OFF) + par * 512 + mblk * 64 + lane;
    unsigned long long xok = 0, seen = 0, raw0 = 0, raw1 = 0;
    int nseq = 0;
    const unsigned long long t0 = __builtin_amdgcn_s_memrealtime();
    for (;;) {
      const unsigned long long wf = __hip_atomic_load(wfl, __ATOMIC_RELAXED, __HIP_MEMORY_SCOPE_SYSTEM);
      if (~xok) {
        const unsigned xf = __hip_atomic_load(xfl, __ATOMIC_RELAXED, __HIP_MEMORY_SCOPE_SYSTEM);
        xok = __ballot((int)(xf - call) >= 0);
      }
      const unsigned w0 = (unsigned)wf, w1 = (unsigned)(wf >> 32);
      unsigned long long rdy = __ballot((int)(w0 - 2u * call) >= 0 && (int)(w1 - 2u * call) >= 0) & xok;
      const unsigned long long fresh = rdy & ~seen;
      if (fresh) {
        seen = rdy;
        const int pos = nseq + __builtin_popcountll(fresh & ((1ull << lane) - 1ull));
        if ((fresh >> lane) & 1ull) *(volatile LAS unsigned char*)(ctrl + C_SEQ + pos) = (unsigned char)lane;
        const unsigned long long m0 = __ballot(w0 & 1u), m1 = __ballot(w1 & 1u);
        unsigned long long rw = (m0 | m1) & fresh;
        if (rw) {
          while (rw) {
            const int st = __builtin_ctzll(rw);
            rw &= rw - 1;
            const int ps = nseq + __builtin_popcountll(fresh & ((1ull << st) - 1ull));
            raw0 |= ((m0 >> st) & 1ull) << ps;
            raw1 |= ((m1 >> st) & 1ull) << ps;
          }
          if (lane == 0) {
            *(volatile LAS unsigned long long*)(ctrl + C_RAWH) = raw0;
            *(volatile LAS unsigned long long*)(ctrl + C_RAWH + 8) = raw1;
            *(volatile LAS unsigned long long*)(ctrl + C_ANY) = raw0 | raw1;
            *(volatile LAS unsigned*)(ctrl + C_ANYRAW) = 1u;
          }
        }
        nseq += __builtin_popcountll(fresh);
        if (lane == 0) *(volatile LAS unsigned*)(ctrl + C_NSEQ) = (unsigned)nseq;
        if (!~rdy) break;
        continue;
      }
      const unsigned long long dt = __builtin_amdgcn_s_memrealtime() - t0;
      if (dt > 20000) {
        if (__builtin_amdgcn_readfirstlane(__hip_atomic_load(tmo, __ATOMIC_RELAXED, __HIP_MEMORY_SCOPE_AGENT)) == call ||
            dt > DEADLINE_TICKS) {
          if (lane == 0) {
            __hip_atomic_store(tmo, call, __ATOMIC_RELAXED, __HIP_MEMORY_SCOPE_AGENT);
            *vfail = 1;
          }
          break;
        }
        __builtin_amdgcn_s_sleep(127);
        __builtin_amdgcn_s_sleep(127);
      } else {
        __builtin_amdgcn_s_sleep(POLL_SLEEP);
      }
    }
    return;
  }

  uint8_t* const slot = myws + SLOT_OFF + (size_t)par * SLOT_BYTES;

  if (wave >= 8) {
    __builtin_amdgcn_s_setprio(HPRIO);
    const int L = (wave - 8) >> 1, half = wave & 1;
    const unsigned ring_addr = (unsigned)(size_t)(smem + L_RING) + (unsigned)L * (RING * SLOTB);
    const unsigned vo = (unsigned)lane * 16u;
    const uint8_t* const xsrc = slot + X_OFF + (size_t)mblk * 4096;
    const uint8_t* const wsrc = slot + (size_t)nblk * WBLK;
    volatile LAS unsigned* const vfull = (volatile LAS unsigned*)(ctrl + C_FULL + (wave - 8) * 4);
    unsigned long long posraw = 0;
    int seqv = 0;
    int j = half, avail = 0, nis = 0, full = 0;
    unsigned dst = ring_addr + (unsigned)half * SLOTB;
    WAIT_VM(0);
    while (full < 16) {
      const v4u ct = *(volatile LAS v4u*)(ctrl + C_ANY);
      const v4u pr = *(volatile LAS v4u*)(ctrl + C_PROG + L * 16);
      const int sv = *(volatile LAS unsigned char*)(ctrl + C_SEQ + lane);
      const bool go = j < avail;
      if (go) {
        const int pos = 2 * j + L;
        const int st = __builtin_amdgcn_readlane(seqv, pos);
        const uint8_t* const xa = xsrc + (size_t)st * 32768;
        const uint8_t* const wa = wsrc + (size_t)st * (32 * WBLK);
        const uint8_t* const wb = wa + 4096;
        unsigned keep;
        asm volatile(
            "s_nop 4\n\t"
            "s_mov_b32 %0, m0\n\t"
            "s_mov_b32 m0, %5\n\t"
            "s_nop 0\n\t"
            "global_load_lds_dwordx4 %1, %2" DMA_HINT "\n\t"
            "global_load_lds_dwordx4 %1, %2 offset:1024" DMA_HINT "\n\t"
            "global_load_lds_dwordx4 %1, %2 offset:2048" DMA_HINT "\n\t"
            "global_load_lds_dwordx4 %1, %2 offset:3072" DMA_HINT "\n\t"
            "s_add_u32 m0, m0, 0x1000\n\t"
            "s_nop 0\n\t"
            "global_load_lds_dwordx4 %1, %3" DMA_HINT "\n\t"
            "global_load_lds_dwordx4 %1, %3 offset:1024" DMA_HINT "\n\t"
            "global_load_lds_dwordx4 %1, %3 offset:2048" DMA_HINT "\n\t"
            "global_load_lds_dwordx4 %1, %3 offset:3072" DMA_HINT "\n\t"
            "s_add_u32 m0, m0, 0x1000\n\t"
            "s_nop 0\n\t"
            "global_load_lds_dwordx4 %1, %4" DMA_HINT "\n\t"
            "global_load_lds_dwordx4 %1, %4 offset:1024" DMA_HINT "\n\t"
            "global_load_lds_dwordx4 %1, %4 offset:2048" DMA_HINT "\n\t"
            "s_mov_b32 m0, %0"
            : "=&s"(keep)
            : "v"(vo), "s"(xa), "s"(wa), "s"(wb), "s"(dst)
            : "memory", "scc");
        if ((posraw >> pos) & 1ull) {
          asm volatile(
              "s_mov_b32 %0, m0\n\t"
              "s_mov_b32 m0, %3\n\t"
              "s_nop 0\n\t"
              "global_load_lds_dwordx4 %1, %2 offset:3072" DMA_HINT "\n\t"
              "s_mov_b32 m0, %0"
              : "=&s"(keep)
              : "v"(vo), "s"(wb), "s"(dst + 8192u)
              : "memory");
        }
        j += 2;
        ++nis;
        dst += 2 * SLOTB;
        if (dst >= ring_addr + RING * SLOTB) dst -= RING * SLOTB;
        if (nis - full == FLY) {
          WAIT_VM(11);
          ++full;
          if (lane == 0) *vfull = (unsigned)full;
        }
      }
      seqv = sv;
      const int lim = (int)min(min(pr[0], pr[1]), min(pr[2], pr[3])) + RING;
      avail = __builtin_amdgcn_readfirstlane(min(lim, ((int)ct[2] + 1 - L) >> 1));
      posraw = uni64(ct[0], ct[1]);
      if (go || j < avail) continue;
      if (nis > full) {
        WAIT_VM(0);
        full = nis;
        if (lane == 0) *vfull = (unsigned)full;
        continue;
      }
      if (__builtin_amdgcn_readfirstlane((int)ct[3])) { if (lane == 0) *vfull = 16u; break; }
      __builtin_amdgcn_s_sleep(LOAD_SLEEP);
    }
    return;
  }

  const int wc = wave & 3, pp = wave >> 2;
  const int g = lane >> 5, c = lane & 31;
  LAS unsigned char* const priv = smem + L_PRIV + wave * 4096;
  *(LAS v4u*)(priv + lane * 16) = v4u{0u, 0u, 0u, 0u};
  *(LAS v4u*)(priv + 1024 + lane * 16) = v4u{0u, 0u, 0u, 0u};
  const float scale = p.sx[0] * p.sw[0];
  {
    unsigned dummy;
    const float* const o = p.out + (size_t)(mblk * 64 + pp * 32 + (lane & 31)) * NN + nblk * 128 + wc * 32;
    asm volatile("global_load_dword %0, %1, off sc1\n\ts_waitcnt vmcnt(0)" : "=&v"(dummy) : "v"(o) : "memory");
  }
  const LAS unsigned char* const ring = smem + L_RING + pp * (RING * SLOTB);
  const int aoff = (2 * g) * 1024 + c * 16;
  const int roff = 4096 + (8 * g) * 512 + wc * 128 + c * 4;
  const int poff = 4096 + (6 * g) * 512 + wc * 128 + c * 4;
  const int loff = 4096 + 12 * 512 + wc * 128 + lane * 2;
  volatile LAS unsigned* const vprog = (volatile LAS unsigned*)(ctrl + C_PROG + wave * 4);

  v16f acc0 = {}, acc1 = {};
  int nfull = 0;
  unsigned long long posraw = 0;
  int so = 0;
  for (int j = 0; j < 32; ++j) {
    if (nfull <= j) {
      for (;;) {
        const unsigned long long f2 = *(volatile LAS unsigned long long*)(ctrl + C_FULL + pp * 8);
        const unsigned anyraw = *(volatile LAS unsigned*)(ctrl + C_ANYRAW);
        nfull = __builtin_amdgcn_readfirstlane((int)min(2u * (unsigned)f2, 2u * (unsigned)(f2 >> 32) + 1u));
        if (nfull > j) {
          if (__builtin_amdgcn_readfirstlane((int)anyraw)) {
            const unsigned long long r = *(volatile LAS unsigned long long*)(ctrl + C_RAWH + (wc >> 1) * 8);
            posraw = uni64((unsigned)r, (unsigned)(r >> 32)) >> pp;
          }
          break;
        }
        __builtin_amdgcn_s_sleep(CONS_SLEEP);
      }
    }
    asm volatile("" ::: "memory");
    const LAS unsigned char* const sl = ring + so;
    v8i a0, a1, bf;
    const unsigned ent = *(const LAS unsigned short*)(sl + loff);
#pragma unroll
    for (int hh = 0; hh < 2; ++hh) {
      const v4u t0v = *(const LAS v4u*)(sl + aoff + hh * 1024);
      const v4u t1v = *(const LAS v4u*)(sl + aoff + hh * 1024 + 512);
#pragma unroll
      for (int e = 0; e < 4; ++e) { a0[hh * 4 + e] = (int)t0v[e]; a1[hh * 4 + e] = (int)t1v[e]; }
    }
    __builtin_amdgcn_sched_barrier(0);
    if ((posraw >> (2 * j)) & 1ull) {
#pragma unroll
      for (int q = 0; q < 8; ++q) bf[q] = *(const LAS int*)(sl + roff + q * 512);
    } else {
      unsigned pa[6];
#pragma unroll
      for (int q = 0; q < 6; ++q) pa[q] = *(const LAS unsigned*)(sl + poff + q * 512);
      const unsigned hdr = __builtin_amdgcn_readfirstlane(ent);
      const unsigned b4 = ((hdr >> 6) & 31u) * 0x04040404u;
      const unsigned ca = (unsigned)(lane - 1) < (hdr & 63u) ? ((ent >> 2) & 2047u) : 2048u + (unsigned)lane * 4u;
      priv[ca] = (unsigned char)((ent & 3u) << 5);
      const v4u c0 = *(const LAS v4u*)(priv + lane * 16);
      const v4u c1 = *(const LAS v4u*)(priv + 1024 + lane * 16);
      priv[ca] = 0;
#pragma unroll
      for (int hh = 0; hh < 2; ++hh) {
        const v4u cc = hh ? c1 : c0;
        const unsigned p0 = pa[3 * hh], p1 = pa[3 * hh + 1], p2 = pa[3 * hh + 2];
        bf[4 * hh + 0] = (int)bfi(0x80808080u, p0, ((p0 & 0x1f1f1f1fu) | cc[0]) + b4);
        bf[4 * hh + 1] = (int)bfi(0x80808080u, p1, ((p1 & 0x1f1f1f1fu) | cc[1]) + b4);
        bf[4 * hh + 2] = (int)bfi(0x80808080u, p2, ((p2 & 0x1f1f1f1fu) | cc[2]) + b4);
        const unsigned u = ((p1 >> 2) & 0x18181818u) | ((p0 >> 4) & 0x06060606u) | ((p2 >> 5) & 0x01010101u) | cc[3];
        bf[4 * hh + 3] = (int)bfi(0x80808080u, p2 << 1, u + b4);
      }
    }
    asm volatile("" ::: "memory");
    if (lane == 0) *vprog = (unsigned)(j + 1);
    acc0 = __builtin_amdgcn_mfma_scale_f32_32x32x64_f8f6f4(a0, bf, acc0, 0, 1, 0, 127, 0, 127);
    acc1 = __builtin_amdgcn_mfma_scale_f32_32x32x64_f8f6f4(a1, bf, acc1, 0, 1, 0, 127, 0, 127);
    so = so + SLOTB == RING * SLOTB ? 0 : so + SLOTB;
  }

  TS(70);
  {
    const v16f give = pp ? acc0 : acc1;
#pragma unroll
    for (int q = 0; q < 4; ++q) {
      v4u v;
#pragma unroll
      for (int e = 0; e < 4; ++e) v[e] = __float_as_uint(give[4 * q + e]);
      *(LAS v4u*)(priv + q * 1024 + lane * 16) = v;
    }
    asm volatile("" ::: "memory");
    if (lane == 0) *(volatile LAS unsigned*)(ctrl + C_DONE + wave * 4) = 1u;
    while (!__builtin_amdgcn_readfirstlane((int)*(volatile LAS unsigned*)(ctrl + C_DONE + (wave ^ 4) * 4)))
      __builtin_amdgcn_s_sleep(1);
    asm volatile("" ::: "memory");
  }
  const bool failed = __builtin_amdgcn_readfirstlane((int)*vfail) != 0;
  LAS unsigned char* const other = smem + L_PRIV + (wave ^ 4) * 4096;
  v4u ov[4];
#pragma unroll
  for (int q = 0; q < 4; ++q) ov[q] = *(const LAS v4u*)(other + q * 1024 + lane * 16);
  float res[16];
#pragma unroll
  for (int r = 0; r < 16; ++r) {
    const float y = ((pp ? acc1[r] : acc0[r]) + __uint_as_float(ov[r >> 2][r & 3])) * scale;
    const float z = fminf(fmaxf(y, -60.0f), 60.0f);
    res[r] = failed ? __builtin_nanf("") : y * __builtin_amdgcn_rcpf(1.0f + __expf(-z));
  }
  asm volatile("" ::: "memory");
#pragma unroll
  for (int r = 0; r < 16; ++r)
    *(LAS float*)(other + ((r & 3) + 8 * (r >> 2) + 4 * g) * 128 + c * 4) = res[r];
  asm volatile("" ::: "memory");
  {
    const auto rsO = __builtin_amdgcn_make_buffer_rsrc((void*)p.out, 0, MS * NN * 4, RSRC3);
    const int oo = ((mblk * 64 + pp * 32 + (lane >> 3)) * NN + nblk * 128 + wc * 32 + (lane & 7) * 4) * 4;
#pragma unroll
    for (int q = 0; q < 4; ++q) {
      const v4u v = *(const LAS v4u*)(other + q * 1024 + lane * 16);
      __builtin_amdgcn_raw_buffer_store_b128(v, rsO, oo + q * 8 * NN * 4, 0, 16);
    }
  }
}

extern "C" void kernel_launch(int rank, int nranks, ncclComm_t comm,
                              hipStream_t stream,
                              void* const* d_in, const int* in_sizes, int n_in,
                              void* d_out, int out_size,
                              const int* in_dtypes, int out_dtype,
                              void* const* d_out_all,
                              void* const* d_ws_all, size_t ws_bytes) {
  (void)comm; (void)in_sizes; (void)n_in; (void)out_size; (void)in_dtypes; (void)out_dtype; (void)d_out_all;
  if (nranks != NR || (unsigned)rank >= (unsigned)NR || ws_bytes < WS_NEED) {
    fprintf(stderr, "kernel_launch: unexpected configuration\n");
    return;
  }
  Params p;
  p.x = (const uint8_t*)d_in[0];
  p.w = (const uint8_t*)d_in[1];
  p.sx = (const float*)d_in[2];
  p.sw = (const float*)d_in[3];
  p.out = (float*)d_out;
  for (int i = 0; i < NR; ++i) p.ws[i] = (uint8_t*)d_ws_all[i];
  p.rank = rank;
  hipLaunchKernelGGL(fused_ag_gemm, dim3(NWG), dim3(NTHREADS), 0, stream, p);
  HIP_CHECK(hipGetLastError());
}
```

```cpp
#include <hip/hip_runtime.h>
#include <rccl/rccl.h>
#include <cstdio>
#include <cstdint>

#define HIP_CHECK(expr) do { hipError_t _e = (expr); if (_e != hipSuccess) { \
    fprintf(stderr, "HIP error %s at %s:%d\n", hipGetErrorString(_e), __FILE__, __LINE__); \
    return; } } while (0)

typedef int      v8i  __attribute__((ext_vector_type(8)));
typedef unsigned v4u  __attribute__((ext_vector_type(4)));
typedef float    v16f __attribute__((ext_vector_type(16)));
#define LAS __attribute__((address_space(3)))

namespace {

constexpr int NR = 8;
constexpr int NN = 4096;
constexpr int MS = 512;
constexpr int KS = 512;
constexpr int NWG = 256;
constexpr int NTHREADS = 960;

constexpr size_t WFLAG_OFF = 0;
constexpr size_t XFLAG_OFF = 32768;
constexpr size_t TMO_OFF   = 40960;
constexpr size_t EPOCH_OFF = 49152;
constexpr size_t SLOT_OFF  = 1u << 20;
constexpr size_t WBLK      = 8192;
constexpr size_t WSRC      = 8 * 32 * WBLK;
constexpr size_t XSRC      = 32 * 512 * 16;
constexpr size_t X_OFF     = NR * WSRC;
constexpr size_t SLOT_BYTES = X_OFF + NR * XSRC;
constexpr size_t WS_NEED   = SLOT_OFF + 2 * SLOT_BYTES;

constexpr int RING = 5;
constexpr int SLOTB = 12288;
constexpr int FLY = 2;
constexpr int L_RING = 0;
constexpr int L_PRIV = 2 * RING * SLOTB;
constexpr int L_SLIST = L_PRIV + 8 * 4096;
constexpr int L_CTRL = L_SLIST + 528;
constexpr int C_ANY = 0;
constexpr int C_NSEQ = 8;
constexpr int C_FAIL = 12;
constexpr int C_RAWH = 16;
constexpr int C_PROG = 32;
constexpr int C_FULL = 64;
constexpr int C_DONE = 80;
constexpr int C_ANYRAW = 176;
constexpr int C_SEQ = 112;
constexpr int L_TOTAL = L_CTRL + 192;

constexpr unsigned long long DEADLINE_TICKS = 50000000ull;
constexpr int AUX_SYS = 17;
constexpr int RSRC3 = 0x00020000;
#define POLL_SLEEP 4
#define FORCE_RAW 0
#define PACE 115
#define SPLIT 1
#define SPLITN 8
#define CONS_SLEEP 1
#define LOAD_SLEEP 1
#define HPRIO 2
#define WIDE_OUT 1
#define TOUCH_OUT 0
#define T_START 90

struct Params {
  const uint8_t* x;
  const uint8_t* w;
  const float* sx;
  const float* sw;
  float* out;
  uint8_t* ws[NR];
  int rank;
};

#define TS(idx) do {} while (0)
#define DMA_HINT ""
#define STR2(x) #x
#define STR(x) STR2(x)
#define WAIT_VM(n) asm volatile("s_waitcnt vmcnt(" STR(n) ")" ::: "memory")

__device__ __forceinline__ v4u transpose4(unsigned a0, unsigned a1, unsigned a2, unsigned a3) {
  const unsigned t0 = __builtin_amdgcn_perm(a1, a0, 0x05010400u);
  const unsigned t1 = __builtin_amdgcn_perm(a1, a0, 0x07030602u);
  const unsigned t2 = __builtin_amdgcn_perm(a3, a2, 0x05010400u);
  const unsigned t3 = __builtin_amdgcn_perm(a3, a2, 0x07030602u);
  v4u r;
  r[0] = __builtin_amdgcn_perm(t2, t0, 0x05040100u);
  r[1] = __builtin_amdgcn_perm(t2, t0, 0x07060302u);
  r[2] = __builtin_amdgcn_perm(t3, t1, 0x05040100u);
  r[3] = __builtin_amdgcn_perm(t3, t1, 0x07060302u);
  return r;
}
__device__ __forceinline__ unsigned bfi(unsigned mask, unsigned a, unsigned b) { return (a & mask) | (b & ~mask); }
__device__ __forceinline__ unsigned long long uni64(unsigned lo, unsigned hi) {
  return ((unsigned long long)(unsigned)__builtin_amdgcn_readfirstlane(hi) << 32) |
         (unsigned)__builtin_amdgcn_readfirstlane(lo);
}

}

__global__ void __launch_bounds__(NTHREADS, 1) fused_ag_gemm(Params p) {
  __shared__ __attribute__((aligned(16))) unsigned char smem_[L_TOTAL];
  LAS unsigned char* const smem = (LAS unsigned char*)smem_;
  LAS unsigned char* const ctrl = smem + L_CTRL;
  const int b = blockIdx.x;
  const int wave = __builtin_amdgcn_readfirstlane(threadIdx.x >> 6);
  const int lane = threadIdx.x & 63;
  const int rank = p.rank;
  uint8_t* const myws = p.ws[rank];

  TS(0);
  unsigned* const ep = (unsigned*)(myws + EPOCH_OFF) + b * 16 + wave;
  const unsigned epv = __hip_atomic_load(ep, __ATOMIC_RELAXED, __HIP_MEMORY_SCOPE_AGENT);
  unsigned* const tmo = (unsigned*)(myws + TMO_OFF);

  if (wave == 12 || wave == 13) {
    const unsigned long long tstart = __builtin_amdgcn_s_memrealtime();
    const int hw = wave - 12;
    const int a = lane & 15, kc = lane >> 4;
    const int dx = b >> 5, ix = b & 31;
    const auto rsW = __builtin_amdgcn_make_buffer_rsrc((void*)p.w, 0, KS * NN, RSRC3);
    const auto rsX = __builtin_amdgcn_make_buffer_rsrc((void*)p.x, 0, 4096 * KS, RSRC3);
    v4u xr[4];
    {
      const int off = (dx * MS + (ix >> 2) * 64 + lane) * KS + (ix & 3) * 128 + hw * 64;
#pragma unroll
      for (int t = 0; t < 4; ++t) xr[t] = __builtin_amdgcn_raw_buffer_load_b128(rsX, off + t * 16, 0, 0);
    }
    unsigned cur[16];
    {
      const int woff = ((b >> 5) * 64 + kc * 16) * NN + (b & 31) * 128 + hw * 64 + a * 4;
#pragma unroll
      for (int t = 0; t < 16; ++t) cur[t] = __builtin_amdgcn_raw_buffer_load_b32(rsW, woff + t * NN, 0, 0);
    }
    __builtin_amdgcn_s_barrier();
    __builtin_amdgcn_s_setprio(3);
    const unsigned call0 = __builtin_amdgcn_readfirstlane(epv);
    const unsigned call = call0 + 1;
    const unsigned par = call0 & 1;
    if (lane == 0) __hip_atomic_store(ep, call, __ATOMIC_RELAXED, __HIP_MEMORY_SCOPE_AGENT);
    const size_t soff = SLOT_OFF + (size_t)par * SLOT_BYTES;
    const unsigned w64 = (unsigned)(ix * 2 + hw);
    auto wait_slot = [&](unsigned ticks) {
      for (;;) {
        const long long left = (long long)(tstart + ticks - __builtin_amdgcn_s_memrealtime());
        if (left <= 0) break;
        if (left > 40) __builtin_amdgcn_s_sleep(10); else __builtin_amdgcn_s_sleep(1);
      }
    };
    uint8_t* const dws = p.ws[dx];
    const bool remote = dx != rank;
    const auto rdX = __builtin_amdgcn_make_buffer_rsrc((void*)(dws + soff + X_OFF + (size_t)rank * XSRC), 0, (int)XSRC, RSRC3);
    const int xoff = (((ix & 3) * 2 + hw) * 8 + (ix >> 2)) * 4096 + lane * 16;
    unsigned* const xflag = (unsigned*)(dws + XFLAG_OFF) + par * 512 + ((ix >> 2) * 8 + rank) * 8 + (ix & 3) * 2 + hw;
    const unsigned xslot = T_START + (w64 * 4u * PACE) / 64u;
    const bool xearly = !remote || w64 < 20u;
    if (xearly) {
      if (remote) wait_slot(xslot);
#pragma unroll
      for (int t = 0; t < 4; ++t) __builtin_amdgcn_raw_buffer_store_b128(xr[t], rdX, xoff + t * 1024, 0, AUX_SYS);
    }

    v4u pk[4];
    unsigned cnt = 0;
    bool raw;
    {
      const unsigned rm = 0x00ff00ffu << (a & 8);
      const unsigned lt = (1u << (lane & 31)) - 1u;
      const unsigned below_lo = lane < 32 ? (rm & lt) : rm;
      const unsigned below_hi = lane < 32 ? 0u : (rm & lt);
      const unsigned posbase = (unsigned)(((kc & 1) << 10) | (((kc >> 1) * 32 + ((4 * a) & 31)) << 4));
      const unsigned lbase = (unsigned)(L_SLIST + (hw * 2 + (a >> 3)) * 128);
      unsigned s = 0;
#pragma unroll
      for (int t = 0; t < 16; t += 4) s = __builtin_amdgcn_sad_u8((cur[t] >> 2) & 0x1f1f1f1fu, 0u, s);
      int tot = 96;
#pragma unroll
      for (int bit = 2; bit < 9; ++bit) tot += __builtin_popcountll(__ballot((s >> bit) & 1u)) << bit;
      int base = (tot - 4291) >> 10;
      base = base < 0 ? 0 : (base > 24 ? 24 : base);
      const unsigned b4 = (unsigned)base * 0x04040404u;
      unsigned ap[16], ex[4];
#pragma unroll
      for (int t = 0; t < 16; ++t) {
        const unsigned dd = (cur[t] | 0x80808080u) - b4;
        const int tt = t & 3;
        const unsigned e2 = tt == 0 ? ((dd >> 5) & 0x03030303u) : tt == 1 ? ((dd >> 3) & 0x0c0c0c0cu)
                          : tt == 2 ? ((dd >> 1) & 0x30303030u) : ((dd << 1) & 0xc0c0c0c0u);
        ex[t >> 2] = tt == 0 ? e2 : (ex[t >> 2] | e2);
        ap[t] = bfi(0x80808080u, cur[t], dd) & 0x9f9f9f9fu;
      }
#pragma unroll
      for (int w = 0; w < 4; ++w) {
        unsigned e = ex[w];
        for (;;) {
          const unsigned long long m = __ballot(e != 0u);
          if (!m) break;
          const unsigned mlo = (unsigned)m, mhi = (unsigned)(m >> 32);
          const unsigned slotn = cnt + 1u + __builtin_popcount(mlo & below_lo) + __builtin_popcount(mhi & below_hi);
          const unsigned bb = (unsigned)__builtin_ctz(e | 0x80000000u) & 30u;
          const unsigned hc = (e >> bb) & 3u;
          const unsigned pos = (unsigned)(4 * w) + posbase + ((bb & 0x18u) << 1) + ((bb >> 1) & 3u);
          const unsigned la = (e != 0u && slotn <= 63u) ? lbase + 2u * slotn : (unsigned)(L_SLIST + 512);
          *(LAS unsigned short*)(smem + la) = (unsigned short)((pos << 2) | hc);
          e ^= hc << bb;
          cnt += __builtin_popcount(mlo & rm) + __builtin_popcount(mhi & rm);
        }
      }
      raw = FORCE_RAW || __ballot(cnt > 63u) != 0ull;
      if (raw) {
#pragma unroll
        for (int q = 0; q < 4; ++q) pk[q] = transpose4(cur[4 * q], cur[4 * q + 1], cur[4 * q + 2], cur[4 * q + 3]);
      } else {
        if ((lane & 55) == 0) *(LAS unsigned short*)(smem + lbase) = (unsigned short)(cnt | ((unsigned)base << 6));
#pragma unroll
        for (int r = 0; r < 4; ++r) {
          const unsigned v = ap[12 + r];
          ap[0 + r] |= (v << 4) & 0x60606060u;
          ap[4 + r] |= (v << 2) & 0x60606060u;
          ap[8 + r] |= ((v >> 1) & 0x40404040u) | ((v << 5) & 0x20202020u);
        }
#pragma unroll
        for (int q = 0; q < 3; ++q) pk[q] = transpose4(ap[4 * q], ap[4 * q + 1], ap[4 * q + 2], ap[4 * q + 3]);
        asm volatile("" ::: "memory");
        pk[3] = *(LAS v4u*)(smem + L_SLIST + hw * 256 + a * 16);
      }
    }
    TS(41);

    const auto rlW = __builtin_amdgcn_make_buffer_rsrc((void*)(myws + soff + (size_t)rank * WSRC), 0, (int)WSRC, RSRC3);
    const auto rdW = __builtin_amdgcn_make_buffer_rsrc((void*)(dws + soff + (size_t)rank * WSRC), 0, (int)WSRC, RSRC3);
    unsigned* const lflags = (unsigned*)(myws + WFLAG_OFF) + par * 4096 + rank * 16;
    unsigned* const dflags = (unsigned*)(dws + WFLAG_OFF) + par * 4096 + rank * 16;
    auto fidx = [&](unsigned it) { return ((it >> 1) & 31u) * 128u + (it >> 6) * 2u + (it & 1u); };
    auto off0 = [&](unsigned it, bool r) { return (int)(it >> 1) * (int)WBLK + kc * (r ? 2048 : 1536) + (int)(it & 1u) * 256 + a * 16; };
    auto off3 = [&](unsigned it, bool r) {
      return r ? off0(it, true) + 1536 : (int)(it >> 1) * (int)WBLK + 6144 + (int)(it & 1u) * 256 + a * 16; };
    unsigned item = (unsigned)b * 2u + (unsigned)hw;
    unsigned fv = call * 2 + (raw ? 1u : 0u);
    {
      const int db = off0(item, raw), db3 = off3(item, raw);
#pragma unroll
      for (int q = 0; q < 3; ++q) __builtin_amdgcn_raw_buffer_store_b128(pk[q], rlW, db + q * 512, 0, AUX_SYS);
      if (raw || (lane < 16 && (unsigned)(lane & 7) * 8u <= cnt))
        __builtin_amdgcn_raw_buffer_store_b128(pk[3], rlW, db3, 0, AUX_SYS);
      WAIT_VM(0);
      if (lane == 0) {
        __hip_atomic_store(lflags + fidx(item), fv, __ATOMIC_RELAXED, __HIP_MEMORY_SCOPE_SYSTEM);
        if (xearly) __hip_atomic_store(xflag, call, __ATOMIC_RELAXED, __HIP_MEMORY_SCOPE_SYSTEM);
      }
      TS(42);
    }
    if (!xearly) {
      wait_slot(xslot);
#pragma unroll
      for (int t = 0; t < 4; ++t) __builtin_amdgcn_raw_buffer_store_b128(xr[t], rdX, xoff + t * 1024, 0, AUX_SYS);
      WAIT_VM(0);
      if (lane == 0) __hip_atomic_store(xflag, call, __ATOMIC_RELAXED, __HIP_MEMORY_SCOPE_SYSTEM);
    }
    if (remote) {
      const unsigned col = (w64 + (unsigned)rank * 8u) & 63u;
      unsigned slot_t = T_START + 4u * PACE + (w64 * (16u * PACE / 5u)) / 64u;
      const unsigned litem = (unsigned)((lane & 7) < dx ? (lane & 7) : (lane & 7) + 1) * 64u + col;
      const unsigned* const lf = lflags + fidx(litem & 511u);
      unsigned lfv = 0;
#pragma unroll 1
      for (int n = 0; n < 8; ++n) {
        const bool r = fv & 1u;
        const int db = off0(item, r), db3 = off3(item, r);
        const unsigned lc = (unsigned)__shfl((int)pk[3][0], lane & 8) & 63u;
        unsigned nitem = 0, nfv = 0;
        v4u n0 = pk[0], n1 = pk[1], n2 = pk[2], n3 = pk[3];
        if (n < 7) {
          nitem = (unsigned)(n < dx ? n : n + 1) * 64u + col;
          while ((int)((nfv = __builtin_amdgcn_readlane(lfv, n)) - 2u * call) < 0) {
            lfv = __hip_atomic_load(lf, __ATOMIC_RELAXED, __HIP_MEMORY_SCOPE_AGENT);
            if ((int)(__builtin_amdgcn_readlane(lfv, n) - 2u * call) >= 0) continue;
            if (__builtin_amdgcn_s_memrealtime() - tstart > DEADLINE_TICKS) {
              if (lane == 0) __hip_atomic_store(tmo, call, __ATOMIC_RELAXED, __HIP_MEMORY_SCOPE_AGENT);
              nfv = 0;
              break;
            }
            __builtin_amdgcn_s_sleep(4);
          }
          asm volatile("" ::: "memory");
          const bool nr = nfv & 1u;
          const int nb0 = off0(nitem, nr), nb3 = off3(nitem, nr);
          n0 = __builtin_amdgcn_raw_buffer_load_b128(rlW, nb0, 0, 0);
          n1 = __builtin_amdgcn_raw_buffer_load_b128(rlW, nb0 + 512, 0, 0);
          n2 = __builtin_amdgcn_raw_buffer_load_b128(rlW, nb0 + 1024, 0, 0);
          n3 = __builtin_amdgcn_raw_buffer_load_b128(rlW, nb3, 0, 0);
        }
        wait_slot(slot_t);
        slot_t += r ? 4u * PACE : 16u * PACE / 5u;
        __builtin_amdgcn_raw_buffer_store_b128(pk[0], rdW, db, 0, AUX_SYS);
        __builtin_amdgcn_raw_buffer_store_b128(pk[1], rdW, db + 512, 0, AUX_SYS);
        if (n < SPLITN) WAIT_VM(1);
        __builtin_amdgcn_raw_buffer_store_b128(pk[2], rdW, db + 1024, 0, AUX_SYS);
        if (r && n < SPLITN) WAIT_VM(1);
        if (r || (lane < 16 && (unsigned)(lane & 7) * 8u <= lc))
          __builtin_amdgcn_raw_buffer_store_b128(pk[3], rdW, db3, 0, AUX_SYS);
        WAIT_VM(0);
        if (lane == 0 && fv) __hip_atomic_store(dflags + fidx(item), fv, __ATOMIC_RELAXED, __HIP_MEMORY_SCOPE_SYSTEM);
        TS(1 + n);
        pk[0] = n0; pk[1] = n1; pk[2] = n2; pk[3] = n3;
        item = nitem;
        fv = nfv;
      }
    }
    TS(40);
    return;
  }

  if (wave == 14 && lane < 12) *(LAS v4u*)(ctrl + lane * 16) = v4u{0u, 0u, 0u, 0u};
  asm volatile("s_waitcnt lgkmcnt(0)" ::: "memory");
  __builtin_amdgcn_s_barrier();

  const unsigned call0 = __builtin_amdgcn_readfirstlane(epv);
  const unsigned call = call0 + 1;
  const unsigned par = call0 & 1;
  if (lane == 0) __hip_atomic_store(ep, call, __ATOMIC_RELAXED, __HIP_MEMORY_SCOPE_AGENT);
  const int nblk = (b & 7) * 4 + ((b >> 3) & 3);
  const int mblk = b >> 5;
  volatile LAS unsigned* const vfail = (volatile LAS unsigned*)(ctrl + C_FAIL);

  if (wave == 14) {
    __builtin_amdgcn_s_setprio(HPRIO);
    const unsigned long long* const wfl = (const unsigned long long*)(myws + WFLAG_OFF) + par * 2048 + nblk * 64 + lane;
    const unsigned* const xfl = (const unsigned*)(myws + XFLAG_OFF) + par * 512 + mblk * 64 + lane;
    unsigned long long xok = 0, seen = 0, raw0 = 0, raw1 = 0;
    int nseq = 0;
    const unsigned long long t0 = __builtin_amdgcn_s_memrealtime();
    for (;;) {
      const unsigned long long wf = __hip_atomic_load(wfl, __ATOMIC_RELAXED, __HIP_MEMORY_SCOPE_SYSTEM);
      if (~xok) {
        const unsigned xf = __hip_atomic_load(xfl, __ATOMIC_RELAXED, __HIP_MEMORY_SCOPE_SYSTEM);
        xok = __ballot((int)(xf - call) >= 0);
      }
      const unsigned w0 = (unsigned)wf, w1 = (unsigned)(wf >> 32);
      unsigned long long rdy = __ballot((int)(w0 - 2u * call) >= 0 && (int)(w1 - 2u * call) >= 0) & xok;
      const unsigned long long fresh = rdy & ~seen;
      if (fresh) {
        seen = rdy;
        const int pos = nseq + __builtin_popcountll(fresh & ((1ull << lane) - 1ull));
        if ((fresh >> lane) & 1ull) *(volatile LAS unsigned char*)(ctrl + C_SEQ + pos) = (unsigned char)lane;
        const unsigned long long m0 = __ballot(w0 & 1u), m1 = __ballot(w1 & 1u);
        unsigned long long rw = (m0 | m1) & fresh;
        if (rw) {
          while (rw) {
            const int st = __builtin_ctzll(rw);
            rw &= rw - 1;
            const int ps = nseq + __builtin_popcountll(fresh & ((1ull << st) - 1ull));
            raw0 |= ((m0 >> st) & 1ull) << ps;
            raw1 |= ((m1 >> st) & 1ull) << ps;
          }
          if (lane == 0) {
            *(volatile LAS unsigned long long*)(ctrl + C_RAWH) = raw0;
            *(volatile LAS unsigned long long*)(ctrl + C_RAWH + 8) = raw1;
            *(volatile LAS unsigned long long*)(ctrl + C_ANY) = raw0 | raw1;
            *(volatile LAS unsigned*)(ctrl + C_ANYRAW) = 1u;
          }
        }
        nseq += __builtin_popcountll(fresh);
        if (lane == 0) *(volatile LAS unsigned*)(ctrl + C_NSEQ) = (unsigned)nseq;
        if (!~rdy) break;
        continue;
      }
      const unsigned long long dt = __builtin_amdgcn_s_memrealtime() - t0;
      if (dt > 20000) {
        if (__builtin_amdgcn_readfirstlane(__hip_atomic_load(tmo, __ATOMIC_RELAXED, __HIP_MEMORY_SCOPE_AGENT)) == call ||
            dt > DEADLINE_TICKS) {
          if (lane == 0) {
            __hip_atomic_store(tmo, call, __ATOMIC_RELAXED, __HIP_MEMORY_SCOPE_AGENT);
            *vfail = 1;
          }
          break;
        }
        __builtin_amdgcn_s_sleep(127);
        __builtin_amdgcn_s_sleep(127);
      } else {
        __builtin_amdgcn_s_sleep(POLL_SLEEP);
      }
    }
    return;
  }

  uint8_t* const slot = myws + SLOT_OFF + (size_t)par * SLOT_BYTES;

  if (wave >= 8) {
    __builtin_amdgcn_s_setprio(HPRIO);
    const int L = (wave - 8) >> 1, half = wave & 1;
    const unsigned ring_addr = (unsigned)(size_t)(smem + L_RING) + (unsigned)L * (RING * SLOTB);
    const unsigned vo = (unsigned)lane * 16u;
    const uint8_t* const xsrc = slot + X_OFF + (size_t)mblk * 4096;
    const uint8_t* const wsrc = slot + (size_t)nblk * WBLK;
    volatile LAS unsigned* const vfull = (volatile LAS unsigned*)(ctrl + C_FULL + (wave - 8) * 4);
    unsigned long long posraw = 0;
    int seqv = 0;
    int j = half, avail = 0, nis = 0, full = 0;
    unsigned dst = ring_addr + (unsigned)half * SLOTB;
    WAIT_VM(0);
    while (full < 16) {
      const v4u ct = *(volatile LAS v4u*)(ctrl + C_ANY);
      const v4u pr = *(volatile LAS v4u*)(ctrl + C_PROG + L * 16);
      const int sv = *(volatile LAS unsigned char*)(ctrl + C_SEQ + lane);
      const bool go = j < avail;
      if (go) {
        const int pos = 2 * j + L;
        const int st = __builtin_amdgcn_readlane(seqv, pos);
        const uint8_t* const xa = xsrc + (size_t)st * 32768;
        const uint8_t* const wa = wsrc + (size_t)st * (32 * WBLK);
        const uint8_t* const wb = wa + 4096;
        unsigned keep;
        asm volatile(
            "s_nop 4\n\t"
            "s_mov_b32 %0, m0\n\t"
            "s_mov_b32 m0, %5\n\t"
            "s_nop 0\n\t"
            "global_load_lds_dwordx4 %1, %2" DMA_HINT "\n\t"
            "global_load_lds_dwordx4 %1, %2 offset:1024" DMA_HINT "\n\t"
            "global_load_lds_dwordx4 %1, %2 offset:2048" DMA_HINT "\n\t"
            "global_load_lds_dwordx4 %1, %2 offset:3072" DMA_HINT "\n\t"
            "s_add_u32 m0, m0, 0x1000\n\t"
            "s_nop 0\n\t"
            "global_load_lds_dwordx4 %1, %3" DMA_HINT "\n\t"
            "global_load_lds_dwordx4 %1, %3 offset:1024" DMA_HINT "\n\t"
            "global_load_lds_dwordx4 %1, %3 offset:2048" DMA_HINT "\n\t"
            "global_load_lds_dwordx4 %1, %3 offset:3072" DMA_HINT "\n\t"
            "s_add_u32 m0, m0, 0x1000\n\t"
            "s_nop 0\n\t"
            "global_load_lds_dwordx4 %1, %4" DMA_HINT "\n\t"
            "global_load_lds_dwordx4 %1, %4 offset:1024" DMA_HINT "\n\t"
            "global_load_lds_dwordx4 %1, %4 offset:2048" DMA_HINT "\n\t"
            "s_mov_b32 m0, %0"
            : "=&s"(keep)
            : "v"(vo), "s"(xa), "s"(wa), "s"(wb), "s"(dst)
            : "memory", "scc");
        if ((posraw >> pos) & 1ull) {
          asm volatile(
              "s_mov_b32 %0, m0\n\t"
              "s_mov_b32 m0, %3\n\t"
              "s_nop 0\n\t"
              "global_load_lds_dwordx4 %1, %2 offset:3072" DMA_HINT "\n\t"
              "s_mov_b32 m0, %0"
              : "=&s"(keep)
              : "v"(vo), "s"(wb), "s"(dst + 8192u)
              : "memory");
        }
        j += 2;
        ++nis;
        dst += 2 * SLOTB;
        if (dst >= ring_addr + RING * SLOTB) dst -= RING * SLOTB;
        if (nis - full == FLY) {
          WAIT_VM(11);
          ++full;
          if (lane == 0) *vfull = (unsigned)full;
        }
      }
      seqv = sv;
      const int lim = (int)min(min(pr[0], pr[1]), min(pr[2], pr[3])) + RING;
      avail = __builtin_amdgcn_readfirstlane(min(lim, ((int)ct[2] + 1 - L) >> 1));
      posraw = uni64(ct[0], ct[1]);
      if (go || j < avail) continue;
      if (nis > full) {
        WAIT_VM(0);
        full = nis;
        if (lane == 0) *vfull = (unsigned)full;
        continue;
      }
      if (__builtin_amdgcn_readfirstlane((int)ct[3])) { if (lane == 0) *vfull = 16u; break; }
      __builtin_amdgcn_s_sleep(LOAD_SLEEP);
    }
    return;
  }

  const int wc = wave & 3, pp = wave >> 2;
  const int g = lane >> 5, c = lane & 31;
  LAS unsigned char* const priv = smem + L_PRIV + wave * 4096;
  *(LAS v4u*)(priv + lane * 16) = v4u{0u, 0u, 0u, 0u};
  *(LAS v4u*)(priv + 1024 + lane * 16) = v4u{0u, 0u, 0u, 0u};
  const float scale = p.sx[0] * p.sw[0];
  const LAS unsigned char* const ring = smem + L_RING + pp * (RING * SLOTB);
  const int aoff = (2 * g) * 1024 + c * 16;
  const int roff = 4096 + (8 * g) * 512 + wc * 128 + c * 4;
  const int poff = 4096 + (6 * g) * 512 + wc * 128 + c * 4;
  const int loff = 4096 + 12 * 512 + wc * 128 + lane * 2;
  volatile LAS unsigned* const vprog = (volatile LAS unsigned*)(ctrl + C_PROG + wave * 4);

  v16f acc0 = {}, acc1 = {};
  int nfull = 0;
  unsigned long long posraw = 0;
  int so = 0;
  for (int j = 0; j < 32; ++j) {
    if (nfull <= j) {
      for (;;) {
        const unsigned long long f2 = *(volatile LAS unsigned long long*)(ctrl + C_FULL + pp * 8);
        const unsigned anyraw = *(volatile LAS unsigned*)(ctrl + C_ANYRAW);
        nfull = __builtin_amdgcn_readfirstlane((int)min(2u * (unsigned)f2, 2u * (unsigned)(f2 >> 32) + 1u));
        if (nfull > j) {
          if (__builtin_amdgcn_readfirstlane((int)anyraw)) {
            const unsigned long long r = *(volatile LAS unsigned long long*)(ctrl + C_RAWH + (wc >> 1) * 8);
            posraw = uni64((unsigned)r, (unsigned)(r >> 32)) >> pp;
          }
          break;
        }
        __builtin_amdgcn_s_sleep(CONS_SLEEP);
      }
    }
    asm volatile("" ::: "memory");
    const LAS unsigned char* const sl = ring + so;
    v8i a0, a1, bf;
    const unsigned ent = *(const LAS unsigned short*)(sl + loff);
#pragma unroll
    for (int hh = 0; hh < 2; ++hh) {
      const v4u t0v = *(const LAS v4u*)(sl + aoff + hh * 1024);
      const v4u t1v = *(const LAS v4u*)(sl + aoff + hh * 1024 + 512);
#pragma unroll
      for (int e = 0; e < 4; ++e) { a0[hh * 4 + e] = (int)t0v[e]; a1[hh * 4 + e] = (int)t1v[e]; }
    }
    __builtin_amdgcn_sched_barrier(0);
    if ((posraw >> (2 * j)) & 1ull) {
#pragma unroll
      for (int q = 0; q < 8; ++q) bf[q] = *(const LAS int*)(sl + roff + q * 512);
    } else {
      unsigned pa[6];
#pragma unroll
      for (int q = 0; q < 6; ++q) pa[q] = *(const LAS unsigned*)(sl + poff + q * 512);
      const unsigned hdr = __builtin_amdgcn_readfirstlane(ent);
      const unsigned b4 = ((hdr >> 6) & 31u) * 0x04040404u;
      const unsigned ca = (unsigned)(lane - 1) < (hdr & 63u) ? ((ent >> 2) & 2047u) : 2048u + (unsigned)lane * 4u;
      priv[ca] = (unsigned char)((ent & 3u) << 5);
      const v4u c0 = *(const LAS v4u*)(priv + lane * 16);
      const v4u c1 = *(const LAS v4u*)(priv + 1024 + lane * 16);
      priv[ca] = 0;
#pragma unroll
      for (int hh = 0; hh < 2; ++hh) {
        const v4u cc = hh ? c1 : c0;
        const unsigned p0 = pa[3 * hh], p1 = pa[3 * hh + 1], p2 = pa[3 * hh + 2];
        bf[4 * hh + 0] = (int)bfi(0x80808080u, p0, ((p0 & 0x1f1f1f1fu) | cc[0]) + b4);
        bf[4 * hh + 1] = (int)bfi(0x80808080u, p1, ((p1 & 0x1f1f1f1fu) | cc[1]) + b4);
        bf[4 * hh + 2] = (int)bfi(0x80808080u, p2, ((p2 & 0x1f1f1f1fu) | cc[2]) + b4);
        const unsigned u = ((p1 >> 2) & 0x18181818u) | ((p0 >> 4) & 0x06060606u) | ((p2 >> 5) & 0x01010101u) | cc[3];
        bf[4 * hh + 3] = (int)bfi(0x80808080u, p2 << 1, u + b4);
      }
    }
    asm volatile("" ::: "memory");
    if (lane == 0) *vprog = (unsigned)(j + 1);
    acc0 = __builtin_amdgcn_mfma_scale_f32_32x32x64_f8f6f4(a0, bf, acc0, 0, 1, 0, 127, 0, 127);
    acc1 = __builtin_amdgcn_mfma_scale_f32_32x32x64_f8f6f4(a1, bf, acc1, 0, 1, 0, 127, 0, 127);
    so = so + SLOTB == RING * SLOTB ? 0 : so + SLOTB;
  }

  TS(70);
  {
    const v16f give = pp ? acc0 : acc1;
#pragma unroll
    for (int q = 0; q < 4; ++q) {
      v4u v;
#pragma unroll
      for (int e = 0; e < 4; ++e) v[e] = __float_as_uint(give[4 * q + e]);
      *(LAS v4u*)(priv + q * 1024 + lane * 16) = v;
    }
    asm volatile("" ::: "memory");
    if (lane == 0) *(volatile LAS unsigned*)(ctrl + C_DONE + wave * 4) = 1u;
    while (!__builtin_amdgcn_readfirstlane((int)*(volatile LAS unsigned*)(ctrl + C_DONE + (wave ^ 4) * 4)))
      __builtin_amdgcn_s_sleep(1);
    asm volatile("" ::: "memory");
  }
  const bool failed = __builtin_amdgcn_readfirstlane((int)*vfail) != 0;
  LAS unsigned char* const other = smem + L_PRIV + (wave ^ 4) * 4096;
  v4u ov[4];
#pragma unroll
  for (int q = 0; q < 4; ++q) ov[q] = *(const LAS v4u*)(other + q * 1024 + lane * 16);
  float res[16];
#pragma unroll
  for (int r = 0; r < 16; ++r) {
    const float y = ((pp ? acc1[r] : acc0[r]) + __uint_as_float(ov[r >> 2][r & 3])) * scale;
    const float z = fminf(fmaxf(y, -60.0f), 60.0f);
    res[r] = failed ? __builtin_nanf("") : y * __builtin_amdgcn_rcpf(1.0f + __expf(-z));
  }
  asm volatile("" ::: "memory");
#pragma unroll
  for (int r = 0; r < 16; ++r)
    *(LAS float*)(other + ((r & 3) + 8 * (r >> 2) + 4 * g) * 128 + c * 4) = res[r];
  asm volatile("" ::: "memory");
  {
    const auto rsO = __builtin_amdgcn_make_buffer_rsrc((void*)p.out, 0, MS * NN * 4, RSRC3);
    const int oo = ((mblk * 64 + pp * 32 + (lane >> 3)) * NN + nblk * 128 + wc * 32 + (lane & 7) * 4) * 4;
#pragma unroll
    for (int q = 0; q < 4; ++q) {
      const v4u v = *(const LAS v4u*)(other + q * 1024 + lane * 16);
      __builtin_amdgcn_raw_buffer_store_b128(v, rsO, oo + q * 8 * NN * 4, 0, 16);
    }
  }
}

extern "C" void kernel_launch(int rank, int nranks, ncclComm_t comm,
                              hipStream_t stream,
                              void* const* d_in, const int* in_sizes, int n_in,
                              void* d_out, int out_size,
                              const int* in_dtypes, int out_dtype,
                              void* const* d_out_all,
                              void* const* d_ws_all, size_t ws_bytes) {
  (void)comm; (void)in_sizes; (void)n_in; (void)out_size; (void)in_dtypes; (void)out_dtype; (void)d_out_all;
  if (nranks != NR || (unsigned)rank >= (unsigned)NR || ws_bytes < WS_NEED) {
    fprintf(stderr, "kernel_launch: unexpected configuration\n");
    return;
  }
  Params p;
  p.x = (const uint8_t*)d_in[0];
  p.w = (const uint8_t*)d_in[1];
  p.sx = (const float*)d_in[2];
  p.sw = (const float*)d_in[3];
  p.out = (float*)d_out;
  for (int i = 0; i < NR; ++i) p.ws[i] = (uint8_t*)d_ws_all[i];
  p.rank = rank;
  hipLaunchKernelGGL(fused_ag_gemm, dim3(NWG), dim3(NTHREADS), 0, stream, p);
  HIP_CHECK(hipGetLastError());
}
```

```cpp
#include <hip/hip_runtime.h>
#include <rccl/rccl.h>
#include <cstdio>
#include <cstdint>

#define HIP_CHECK(expr) do { hipError_t _e = (expr); if (_e != hipSuccess) { \
    fprintf(stderr, "HIP error %s at %s:%d\n", hipGetErrorString(_e), __FILE__, __LINE__); \
    return; } } while (0)

typedef int      v8i  __attribute__((ext_vector_type(8)));
typedef unsigned v4u  __attribute__((ext_vector_type(4)));
typedef float    v16f __attribute__((ext_vector_type(16)));
#define LAS __attribute__((address_space(3)))

namespace {

constexpr int NR = 8;
constexpr int NN = 4096;
constexpr int MS = 512;
constexpr int KS = 512;
constexpr int NWG = 256;
constexpr int NTHREADS = 960;

constexpr size_t WFLAG_OFF = 0;
constexpr size_t XFLAG_OFF = 32768;
constexpr size_t TMO_OFF   = 40960;
constexpr size_t EPOCH_OFF = 49152;
constexpr size_t SLOT_OFF  = 1u << 20;
constexpr size_t WBLK      = 8192;
constexpr size_t WSRC      = 8 * 32 * WBLK;
constexpr size_t XSRC      = 32 * 512 * 16;
constexpr size_t X_OFF     = NR * WSRC;
constexpr size_t SLOT_BYTES = X_OFF + NR * XSRC;
constexpr size_t WS_NEED   = SLOT_OFF + 2 * SLOT_BYTES;

constexpr int RING = 5;
constexpr int SLOTB = 12288;
constexpr int FLY = 2;
constexpr int L_RING = 0;
constexpr int L_PRIV = 2 * RING * SLOTB;
constexpr int L_SLIST = L_PRIV + 8 * 4096;
constexpr int L_CTRL = L_SLIST + 528;
constexpr int C_ANY = 0;
constexpr int C_NSEQ = 8;
constexpr int C_FAIL = 12;
constexpr int C_RAWH = 16;
constexpr int C_PROG = 32;
constexpr int C_FULL = 64;
constexpr int C_DONE = 80;
constexpr int C_ANYRAW = 176;
constexpr int C_SEQ = 112;
constexpr int L_TOTAL = L_CTRL + 192;

constexpr unsigned long long DEADLINE_TICKS = 50000000ull;
constexpr int AUX_SYS = 17;
constexpr int RSRC3 = 0x00020000;
constexpr unsigned PACE = 116;
constexpr unsigned T_START = 10;
constexpr unsigned XEARLY = 20;

struct Params {
  const uint8_t* x;
  const uint8_t* w;
  const float* sx;
  const float* sw;
  float* out;
  uint8_t* ws[NR];
  int rank;
};

#define WAIT_VM(n) asm volatile("s_waitcnt vmcnt(" #n ")" ::: "memory")

__device__ __forceinline__ v4u transpose4(unsigned a0, unsigned a1, unsigned a2, unsigned a3) {
  const unsigned t0 = __builtin_amdgcn_perm(a1, a0, 0x05010400u);
  const unsigned t1 = __builtin_amdgcn_perm(a1, a0, 0x07030602u);
  const unsigned t2 = __builtin_amdgcn_perm(a3, a2, 0x05010400u);
  const unsigned t3 = __builtin_amdgcn_perm(a3, a2, 0x07030602u);
  v4u r;
  r[0] = __builtin_amdgcn_perm(t2, t0, 0x05040100u);
  r[1] = __builtin_amdgcn_perm(t2, t0, 0x07060302u);
  r[2] = __builtin_amdgcn_perm(t3, t1, 0x05040100u);
  r[3] = __builtin_amdgcn_perm(t3, t1, 0x07060302u);
  return r;
}
__device__ __forceinline__ unsigned bfi(unsigned mask, unsigned a, unsigned b) { return (a & mask) | (b & ~mask); }
__device__ __forceinline__ unsigned long long uni64(unsigned lo, unsigned hi) {
  return ((unsigned long long)(unsigned)__builtin_amdgcn_readfirstlane(hi) << 32) |
         (unsigned)__builtin_amdgcn_readfirstlane(lo);
}

}

__global__ void __launch_bounds__(NTHREADS, 1) fused_ag_gemm(Params p) {
  __shared__ __attribute__((aligned(16))) unsigned char smem_[L_TOTAL];
  LAS unsigned char* const smem = (LAS unsigned char*)smem_;
  LAS unsigned char* const ctrl = smem + L_CTRL;
  const int b = blockIdx.x;
  const int wave = __builtin_amdgcn_readfirstlane(threadIdx.x >> 6);
  const int lane = threadIdx.x & 63;
  const int rank = p.rank;
  uint8_t* const myws = p.ws[rank];
  unsigned* const ep = (unsigned*)(myws + EPOCH_OFF) + b * 16 + wave;
  const unsigned epv = __hip_atomic_load(ep, __ATOMIC_RELAXED, __HIP_MEMORY_SCOPE_AGENT);
  unsigned* const tmo = (unsigned*)(myws + TMO_OFF);

  if (wave == 12 || wave == 13) {
    const unsigned long long tstart = __builtin_amdgcn_s_memrealtime();
    const int hw = wave - 12;
    const int a = lane & 15, kc = lane >> 4;
    const int dx = b >> 5, ix = b & 31;
    const auto rsW = __builtin_amdgcn_make_buffer_rsrc((void*)p.w, 0, KS * NN, RSRC3);
    const auto rsX = __builtin_amdgcn_make_buffer_rsrc((void*)p.x, 0, 4096 * KS, RSRC3);
    v4u xr[4];
    {
      const int off = (dx * MS + (ix >> 2) * 64 + lane) * KS + (ix & 3) * 128 + hw * 64;
#pragma unroll
      for (int t = 0; t < 4; ++t) xr[t] = __builtin_amdgcn_raw_buffer_load_b128(rsX, off + t * 16, 0, 0);
    }
    unsigned cur[16];
    {
      const int woff = ((b >> 5) * 64 + kc * 16) * NN + (b & 31) * 128 + hw * 64 + a * 4;
#pragma unroll
      for (int t = 0; t < 16; ++t) cur[t] = __builtin_amdgcn_raw_buffer_load_b32(rsW, woff + t * NN, 0, 0);
    }
    __builtin_amdgcn_s_barrier();
    __builtin_amdgcn_s_setprio(3);
    const unsigned call0 = __builtin_amdgcn_readfirstlane(epv);
    const unsigned call = call0 + 1;
    const unsigned par = call0 & 1;
    if (lane == 0) __hip_atomic_store(ep, call, __ATOMIC_RELAXED, __HIP_MEMORY_SCOPE_AGENT);
    const size_t soff = SLOT_OFF + (size_t)par * SLOT_BYTES;
    const unsigned w64 = (unsigned)(ix * 2 + hw);
    auto wait_slot = [&](unsigned ticks) {
      for (;;) {
        const long long left = (long long)(tstart + ticks - __builtin_amdgcn_s_memrealtime());
        if (left <= 0) break;
        if (left > 40) __builtin_amdgcn_s_sleep(10); else __builtin_amdgcn_s_sleep(1);
      }
    };
    uint8_t* const dws = p.ws[dx];
    const bool remote = dx != rank;
    const auto rdX = __builtin_amdgcn_make_buffer_rsrc((void*)(dws + soff + X_OFF + (size_t)rank * XSRC), 0, (int)XSRC, RSRC3);
    const int xoff = (((ix & 3) * 2 + hw) * 8 + (ix >> 2)) * 4096 + lane * 16;
    unsigned* const xflag = (unsigned*)(dws + XFLAG_OFF) + par * 512 + ((ix >> 2) * 8 + rank) * 8 + (ix & 3) * 2 + hw;
    const unsigned xslot = T_START + (w64 * 4u * PACE) / 64u;
    const bool xearly = !remote || w64 < XEARLY;
    if (xearly) {
      if (remote) wait_slot(xslot);
#pragma unroll
      for (int t = 0; t < 4; ++t) __builtin_amdgcn_raw_buffer_store_b128(xr[t], rdX, xoff + t * 1024, 0, AUX_SYS);
    }

    v4u pk[4];
    unsigned cnt = 0;
    bool raw;
    {
      const unsigned rm = 0x00ff00ffu << (a & 8);
      const unsigned lt = (1u << (lane & 31)) - 1u;
      const unsigned below_lo = lane < 32 ? (rm & lt) : rm;
      const unsigned below_hi = lane < 32 ? 0u : (rm & lt);
      const unsigned posbase = (unsigned)(((kc & 1) << 10) | (((kc >> 1) * 32 + ((4 * a) & 31)) << 4));
      const unsigned lbase = (unsigned)(L_SLIST + (hw * 2 + (a >> 3)) * 128);
      unsigned s = 0;
#pragma unroll
      for (int t = 0; t < 16; t += 4) s = __builtin_amdgcn_sad_u8((cur[t] >> 2) & 0x1f1f1f1fu, 0u, s);
      int tot = 96;
#pragma unroll
      for (int bit = 2; bit < 9; ++bit) tot += __builtin_popcountll(__ballot((s >> bit) & 1u)) << bit;
      int base = (tot - 4291) >> 10;
      base = base < 0 ? 0 : (base > 24 ? 24 : base);
      const unsigned b4 = (unsigned)base * 0x04040404u;
      unsigned ap[16], ex[4];
#pragma unroll
      for (int t = 0; t < 16; ++t) {
        const unsigned dd = (cur[t] | 0x80808080u) - b4;
        const int tt = t & 3;
        const unsigned e2 = tt == 0 ? ((dd >> 5) & 0x03030303u) : tt == 1 ? ((dd >> 3) & 0x0c0c0c0cu)
                          : tt == 2 ? ((dd >> 1) & 0x30303030u) : ((dd << 1) & 0xc0c0c0c0u);
        ex[t >> 2] = tt == 0 ? e2 : (ex[t >> 2] | e2);
        ap[t] = bfi(0x80808080u, cur[t], dd) & 0x9f9f9f9fu;
      }
      unsigned nex = 0;
#pragma unroll
      for (int w = 0; w < 4; ++w) nex += __builtin_popcount((ex[w] | (ex[w] >> 1)) & 0x55555555u);
      const bool hopeless = __ballot(nex >= 16u) != 0ull;
#pragma unroll
      for (int w = 0; w < 4; ++w) {
        unsigned e = hopeless ? 0u : ex[w];
        for (;;) {
          const unsigned long long m = __ballot(e != 0u);
          if (!m) break;
          const unsigned mlo = (unsigned)m, mhi = (unsigned)(m >> 32);
          const unsigned slotn = cnt + 1u + __builtin_popcount(mlo & below_lo) + __builtin_popcount(mhi & below_hi);
          const unsigned bb = (unsigned)__builtin_ctz(e | 0x80000000u) & 30u;
          const unsigned hc = (e >> bb) & 3u;
          const unsigned pos = (unsigned)(4 * w) + posbase + ((bb & 0x18u) << 1) + ((bb >> 1) & 3u);
          const unsigned la = (e != 0u && slotn <= 63u) ? lbase + 2u * slotn : (unsigned)(L_SLIST + 512);
          *(LAS unsigned short*)(smem + la) = (unsigned short)((pos << 2) | hc);
          e ^= hc << bb;
          cnt += __builtin_popcount(mlo & rm) + __builtin_popcount(mhi & rm);
        }
      }
      raw = hopeless || __ballot(cnt > 63u) != 0ull;
      if (raw) {
#pragma unroll
        for (int q = 0; q < 4; ++q) pk[q] = transpose4(cur[4 * q], cur[4 * q + 1], cur[4 * q + 2], cur[4 * q + 3]);
      } else {
        if ((lane & 55) == 0) *(LAS unsigned short*)(smem + lbase) = (unsigned short)(cnt | ((unsigned)base << 6));
#pragma unroll
        for (int r = 0; r < 4; ++r) {
          const unsigned v = ap[12 + r];
          ap[0 + r] |= (v << 4) & 0x60606060u;
          ap[4 + r] |= (v << 2) & 0x60606060u;
          ap[8 + r] |= ((v >> 1) & 0x40404040u) | ((v << 5) & 0x20202020u);
        }
#pragma unroll
        for (int q = 0; q < 3; ++q) pk[q] = transpose4(ap[4 * q], ap[4 * q + 1], ap[4 * q + 2], ap[4 * q + 3]);
        asm volatile("" ::: "memory");
        pk[3] = *(LAS v4u*)(smem + L_SLIST + hw * 256 + a * 16);
      }
    }
    const auto rlW = __builtin_amdgcn_make_buffer_rsrc((void*)(myws + soff + (size_t)rank * WSRC), 0, (int)WSRC, RSRC3);
    const auto rdW = __builtin_amdgcn_make_buffer_rsrc((void*)(dws + soff + (size_t)rank * WSRC), 0, (int)WSRC, RSRC3);
    unsigned* const lflags = (unsigned*)(myws + WFLAG_OFF) + par * 4096 + rank * 16;
    unsigned* const dflags = (unsigned*)(dws + WFLAG_OFF) + par * 4096 + rank * 16;
    auto fidx = [&](unsigned it) { return ((it >> 1) & 31u) * 128u + (it >> 6) * 2u + (it & 1u); };
    auto off0 = [&](unsigned it, bool r) { return (int)(it >> 1) * (int)WBLK + kc * (r ? 2048 : 1536) + (int)(it & 1u) * 256 + a * 16; };
    auto off3 = [&](unsigned it, bool r) {
      return r ? off0(it, true) + 1536 : (int)(it >> 1) * (int)WBLK + 6144 + (int)(it & 1u) * 256 + a * 16; };
    unsigned item = (unsigned)b * 2u + (unsigned)hw;
    unsigned fv = call * 2 + (raw ? 1u : 0u);
    {
      const int db = off0(item, raw), db3 = off3(item, raw);
#pragma unroll
      for (int q = 0; q < 3; ++q) __builtin_amdgcn_raw_buffer_store_b128(pk[q], rlW, db + q * 512, 0, AUX_SYS);
      if (raw || (lane < 16 && (unsigned)(lane & 7) * 8u <= cnt))
        __builtin_amdgcn_raw_buffer_store_b128(pk[3], rlW, db3, 0, AUX_SYS);
      WAIT_VM(0);
      if (lane == 0) {
        __hip_atomic_store(lflags + fidx(item), fv, __ATOMIC_RELAXED, __HIP_MEMORY_SCOPE_SYSTEM);
        if (xearly) __hip_atomic_store(xflag, call, __ATOMIC_RELAXED, __HIP_MEMORY_SCOPE_SYSTEM);
      }
    }
    if (!xearly) {
      wait_slot(xslot);
#pragma unroll
      for (int t = 0; t < 4; ++t) __builtin_amdgcn_raw_buffer_store_b128(xr[t], rdX, xoff + t * 1024, 0, AUX_SYS);
      WAIT_VM(0);
      if (lane == 0) __hip_atomic_store(xflag, call, __ATOMIC_RELAXED, __HIP_MEMORY_SCOPE_SYSTEM);
    }
    if (remote) {
      const unsigned col = (w64 + (unsigned)rank * 8u) & 63u;
      unsigned slot_t = T_START + 4u * PACE + (w64 * (16u * PACE / 5u)) / 64u;
      const unsigned litem = (unsigned)((lane & 7) < dx ? (lane & 7) : (lane & 7) + 1) * 64u + col;
      const unsigned* const lf = lflags + fidx(litem & 511u);
      unsigned lfv = 0;
#pragma unroll 1
      for (int n = 0; n < 8; ++n) {
        const bool r = fv & 1u;
        const int db = off0(item, r), db3 = off3(item, r);
        const unsigned lc = (unsigned)__shfl((int)pk[3][0], lane & 8) & 63u;
        unsigned nitem = 0, nfv = 0;
        v4u n0 = pk[0], n1 = pk[1], n2 = pk[2], n3 = pk[3];
        if (n < 7) {
          nitem = (unsigned)(n < dx ? n : n + 1) * 64u + col;
          while ((int)((nfv = __builtin_amdgcn_readlane(lfv, n)) - 2u * call) < 0) {
            lfv = __hip_atomic_load(lf, __ATOMIC_RELAXED, __HIP_MEMORY_SCOPE_AGENT);
            if ((int)(__builtin_amdgcn_readlane(lfv, n) - 2u * call) >= 0) continue;
            if (__builtin_amdgcn_s_memrealtime() - tstart > DEADLINE_TICKS) {
              if (lane == 0) __hip_atomic_store(tmo, call, __ATOMIC_RELAXED, __HIP_MEMORY_SCOPE_AGENT);
              nfv = 0;
              break;
            }
            __builtin_amdgcn_s_sleep(4);
          }
          asm volatile("" ::: "memory");
          const bool nr = nfv & 1u;
          const int nb0 = off0(nitem, nr), nb3 = off3(nitem, nr);
          n0 = __builtin_amdgcn_raw_buffer_load_b128(rlW, nb0, 0, 0);
          n1 = __builtin_amdgcn_raw_buffer_load_b128(rlW, nb0 + 512, 0, 0);
          n2 = __builtin_amdgcn_raw_buffer_load_b128(rlW, nb0 + 1024, 0, 0);
          n3 = __builtin_amdgcn_raw_buffer_load_b128(rlW, nb3, 0, 0);
        }
        wait_slot(slot_t);
        slot_t += r ? 4u * PACE : 16u * PACE / 5u;
        __builtin_amdgcn_raw_buffer_store_b128(pk[0], rdW, db, 0, AUX_SYS);
        __builtin_amdgcn_raw_buffer_store_b128(pk[1], rdW, db + 512, 0, AUX_SYS);
        WAIT_VM(1);
        __builtin_amdgcn_raw_buffer_store_b128(pk[2], rdW, db + 1024, 0, AUX_SYS);
        if (r) WAIT_VM(1);
        if (r || (lane < 16 && (unsigned)(lane & 7) * 8u <= lc))
          __builtin_amdgcn_raw_buffer_store_b128(pk[3], rdW, db3, 0, AUX_SYS);
        WAIT_VM(0);
        if (lane == 0 && fv) __hip_atomic_store(dflags + fidx(item), fv, __ATOMIC_RELAXED, __HIP_MEMORY_SCOPE_SYSTEM);
        pk[0] = n0; pk[1] = n1; pk[2] = n2; pk[3] = n3;
        item = nitem;
        fv = nfv;
      }
    }
    return;
  }

  if (wave == 14 && lane < 12) *(LAS v4u*)(ctrl + lane * 16) = v4u{0u, 0u, 0u, 0u};
  asm volatile("s_waitcnt lgkmcnt(0)" ::: "memory");
  __builtin_amdgcn_s_barrier();

  const unsigned call0 = __builtin_amdgcn_readfirstlane(epv);
  const unsigned call = call0 + 1;
  const unsigned par = call0 & 1;
  if (lane == 0) __hip_atomic_store(ep, call, __ATOMIC_RELAXED, __HIP_MEMORY_SCOPE_AGENT);
  const int nblk = (b & 7) * 4 + ((b >> 3) & 3);
  const int mblk = b >> 5;
  volatile LAS unsigned* const vfail = (volatile LAS unsigned*)(ctrl + C_FAIL);

  if (wave == 14) {
    __builtin_amdgcn_s_setprio(2);
    const unsigned long long* const wfl = (const unsigned long long*)(myws + WFLAG_OFF) + par * 2048 + nblk * 64 + lane;
    const unsigned* const xfl = (const unsigned*)(myws + XFLAG_OFF) + par * 512 + mblk * 64 + lane;
    unsigned long long xok = 0, seen = 0, raw0 = 0, raw1 = 0;
    int nseq = 0;
    const unsigned long long t0 = __builtin_amdgcn_s_memrealtime();
    for (;;) {
      const unsigned long long wf = __hip_atomic_load(wfl, __ATOMIC_RELAXED, __HIP_MEMORY_SCOPE_SYSTEM);
      if (~xok) {
        const unsigned xf = __hip_atomic_load(xfl, __ATOMIC_RELAXED, __HIP_MEMORY_SCOPE_SYSTEM);
        xok = __ballot((int)(xf - call) >= 0);
      }
      const unsigned w0 = (unsigned)wf, w1 = (unsigned)(wf >> 32);
      unsigned long long rdy = __ballot((int)(w0 - 2u * call) >= 0 && (int)(w1 - 2u * call) >= 0) & xok;
      const unsigned long long fresh = rdy & ~seen;
      if (fresh) {
        seen = rdy;
        const int pos = nseq + __builtin_popcountll(fresh & ((1ull << lane) - 1ull));
        if ((fresh >> lane) & 1ull) *(volatile LAS unsigned char*)(ctrl + C_SEQ + pos) = (unsigned char)lane;
        const unsigned long long m0 = __ballot(w0 & 1u), m1 = __ballot(w1 & 1u);
        unsigned long long rw = (m0 | m1) & fresh;
        if (rw) {
          while (rw) {
            const int st = __builtin_ctzll(rw);
            rw &= rw - 1;
            const int ps = nseq + __builtin_popcountll(fresh & ((1ull << st) - 1ull));
            raw0 |= ((m0 >> st) & 1ull) << ps;
            raw1 |= ((m1 >> st) & 1ull) << ps;
          }
          if (lane == 0) {
            *(volatile LAS unsigned long long*)(ctrl + C_RAWH) = raw0;
            *(volatile LAS unsigned long long*)(ctrl + C_RAWH + 8) = raw1;
            *(volatile LAS unsigned long long*)(ctrl + C_ANY) = raw0 | raw1;
            *(volatile LAS unsigned*)(ctrl + C_ANYRAW) = 1u;
          }
        }
        nseq += __builtin_popcountll(fresh);
        if (lane == 0) *(volatile LAS unsigned*)(ctrl + C_NSEQ) = (unsigned)nseq;
        if (!~rdy) break;
        continue;
      }
      const unsigned long long dt = __builtin_amdgcn_s_memrealtime() - t0;
      if (dt > 20000) {
        if (__builtin_amdgcn_readfirstlane(__hip_atomic_load(tmo, __ATOMIC_RELAXED, __HIP_MEMORY_SCOPE_AGENT)) == call ||
            dt > DEADLINE_TICKS) {
          if (lane == 0) {
            __hip_atomic_store(tmo, call, __ATOMIC_RELAXED, __HIP_MEMORY_SCOPE_AGENT);
            *vfail = 1;
          }
          break;
        }
        __builtin_amdgcn_s_sleep(127);
        __builtin_amdgcn_s_sleep(127);
      } else {
        __builtin_amdgcn_s_sleep(4);
      }
    }
    return;
  }

  uint8_t* const slot = myws + SLOT_OFF + (size_t)par * SLOT_BYTES;

  if (wave >= 8) {
    __builtin_amdgcn_s_setprio(2);
    const int L = (wave - 8) >> 1, half = wave & 1;
    const unsigned ring_addr = (unsigned)(size_t)(smem + L_RING) + (unsigned)L * (RING * SLOTB);
    const unsigned vo = (unsigned)lane * 16u;
    const uint8_t* const xsrc = slot + X_OFF + (size_t)mblk * 4096;
    const uint8_t* const wsrc = slot + (size_t)nblk * WBLK;
    volatile LAS unsigned* const vfull = (volatile LAS unsigned*)(ctrl + C_FULL + (wave - 8) * 4);
    unsigned long long posraw = 0;
    int seqv = 0;
    int j = half, avail = 0, nis = 0, full = 0;
    unsigned dst = ring_addr + (unsigned)half * SLOTB;
    WAIT_VM(0);
    while (full < 16) {
      const v4u ct = *(volatile LAS v4u*)(ctrl + C_ANY);
      const v4u pr = *(volatile LAS v4u*)(ctrl + C_PROG + L * 16);
      const int sv = *(volatile LAS unsigned char*)(ctrl + C_SEQ + lane);
      const bool go = j < avail;
      if (go) {
        const int pos = 2 * j + L;
        const int st = __builtin_amdgcn_readlane(seqv, pos);
        const uint8_t* const xa = xsrc + (size_t)st * 32768;
        const uint8_t* const wa = wsrc + (size_t)st * (32 * WBLK);
        const uint8_t* const wb = wa + 4096;
        unsigned keep;
        asm volatile(
            "s_nop 4\n\t"
            "s_mov_b32 %0, m0\n\t"
            "s_mov_b32 m0, %5\n\t"
            "s_nop 0\n\t"
            "global_load_lds_dwordx4 %1, %2\n\t"
            "global_load_lds_dwordx4 %1, %2 offset:1024\n\t"
            "global_load_lds_dwordx4 %1, %2 offset:2048\n\t"
            "global_load_lds_dwordx4 %1, %2 offset:3072\n\t"
            "s_add_u32 m0, m0, 0x1000\n\t"
            "s_nop 0\n\t"
            "global_load_lds_dwordx4 %1, %3\n\t"
            "global_load_lds_dwordx4 %1, %3 offset:1024\n\t"
            "global_load_lds_dwordx4 %1, %3 offset:2048\n\t"
            "global_load_lds_dwordx4 %1, %3 offset:3072\n\t"
            "s_add_u32 m0, m0, 0x1000\n\t"
            "s_nop 0\n\t"
            "global_load_lds_dwordx4 %1, %4\n\t"
            "global_load_lds_dwordx4 %1, %4 offset:1024\n\t"
            "global_load_lds_dwordx4 %1, %4 offset:2048\n\t"
            "s_mov_b32 m0, %0"
            : "=&s"(keep)
            : "v"(vo), "s"(xa), "s"(wa), "s"(wb), "s"(dst)
            : "memory", "scc");
        if ((posraw >> pos) & 1ull) {
          asm volatile(
              "s_mov_b32 %0, m0\n\t"
              "s_mov_b32 m0, %3\n\t"
              "s_nop 0\n\t"
              "global_load_lds_dwordx4 %1, %2 offset:3072\n\t"
              "s_mov_b32 m0, %0"
              : "=&s"(keep)
              : "v"(vo), "s"(wb), "s"(dst + 8192u)
              : "memory");
        }
        j += 2;
        ++nis;
        dst += 2 * SLOTB;
        if (dst >= ring_addr + RING * SLOTB) dst -= RING * SLOTB;
        if (nis - full == FLY) {
          WAIT_VM(11);
          ++full;
          if (lane == 0) *vfull = (unsigned)full;
        }
      }
      seqv = sv;
      const int lim = (int)min(min(pr[0], pr[1]), min(pr[2], pr[3])) + RING;
      avail = __builtin_amdgcn_readfirstlane(min(lim, ((int)ct[2] + 1 - L) >> 1));
      posraw = uni64(ct[0], ct[1]);
      if (go || j < avail) continue;
      if (nis > full) {
        WAIT_VM(0);
        full = nis;
        if (lane == 0) *vfull = (unsigned)full;
        continue;
      }
      if (__builtin_amdgcn_readfirstlane((int)ct[3])) { if (lane == 0) *vfull = 16u; break; }
      __builtin_amdgcn_s_sleep(1);
    }
    return;
  }

  const int wc = wave & 3, pp = wave >> 2;
  const int g = lane >> 5, c = lane & 31;
  LAS unsigned char* const priv = smem + L_PRIV + wave * 4096;
  *(LAS v4u*)(priv + lane * 16) = v4u{0u, 0u, 0u, 0u};
  *(LAS v4u*)(priv + 1024 + lane * 16) = v4u{0u, 0u, 0u, 0u};
  const float scale = p.sx[0] * p.sw[0];
  const LAS unsigned char* const ring = smem + L_RING + pp * (RING * SLOTB);
  const int aoff = (2 * g) * 1024 + c * 16;
  const int roff = 4096 + (8 * g) * 512 + wc * 128 + c * 4;
  const int poff = 4096 + (6 * g) * 512 + wc * 128 + c * 4;
  const int loff = 4096 + 12 * 512 + wc * 128 + lane * 2;
  volatile LAS unsigned* const vprog = (volatile LAS unsigned*)(ctrl + C_PROG + wave * 4);

  v16f acc0 = {}, acc1 = {};
  int nfull = 0;
  unsigned long long posraw = 0;
  int so = 0;
  for (int j = 0; j < 32; ++j) {
    if (nfull <= j) {
      for (;;) {
        const unsigned long long f2 = *(volatile LAS unsigned long long*)(ctrl + C_FULL + pp * 8);
        const unsigned anyraw = *(volatile LAS unsigned*)(ctrl + C_ANYRAW);
        nfull = __builtin_amdgcn_readfirstlane((int)min(2u * (unsigned)f2, 2u * (unsigned)(f2 >> 32) + 1u));
        if (nfull > j) {
          if (__builtin_amdgcn_readfirstlane((int)anyraw)) {
            const unsigned long long r = *(volatile LAS unsigned long long*)(ctrl + C_RAWH + (wc >> 1) * 8);
            posraw = uni64((unsigned)r, (unsigned)(r >> 32)) >> pp;
          }
          break;
        }
        __builtin_amdgcn_s_sleep(1);
      }
    }
    asm volatile("" ::: "memory");
    const LAS unsigned char* const sl = ring + so;
    v8i a0, a1, bf;
    const unsigned ent = *(const LAS unsigned short*)(sl + loff);
#pragma unroll
    for (int hh = 0; hh < 2; ++hh) {
      const v4u t0v = *(const LAS v4u*)(sl + aoff + hh * 1024);
      const v4u t1v = *(const LAS v4u*)(sl + aoff + hh * 1024 + 512);
#pragma unroll
      for (int e = 0; e < 4; ++e) { a0[hh * 4 + e] = (int)t0v[e]; a1[hh * 4 + e] = (int)t1v[e]; }
    }
    __builtin_amdgcn_sched_barrier(0);
    if ((posraw >> (2 * j)) & 1ull) {
#pragma unroll
      for (int q = 0; q < 8; ++q) bf[q] = *(const LAS int*)(sl + roff + q * 512);
    } else {
      unsigned pa[6];
#pragma unroll
      for (int q = 0; q < 6; ++q) pa[q] = *(const LAS unsigned*)(sl + poff + q * 512);
      const unsigned hdr = __builtin_amdgcn_readfirstlane(ent);
      const unsigned b4 = ((hdr >> 6) & 31u) * 0x04040404u;
      const unsigned ca = (unsigned)(lane - 1) < (hdr & 63u) ? ((ent >> 2) & 2047u) : 2048u + (unsigned)lane * 4u;
      priv[ca] = (unsigned char)((ent & 3u) << 5);
      const v4u c0 = *(const LAS v4u*)(priv + lane * 16);
      const v4u c1 = *(const LAS v4u*)(priv + 1024 + lane * 16);
      priv[ca] = 0;
#pragma unroll
      for (int hh = 0; hh < 2; ++hh) {
        const v4u cc = hh ? c1 : c0;
        const unsigned p0 = pa[3 * hh], p1 = pa[3 * hh + 1], p2 = pa[3 * hh + 2];
        bf[4 * hh + 0] = (int)bfi(0x80808080u, p0, ((p0 & 0x1f1f1f1fu) | cc[0]) + b4);
        bf[4 * hh + 1] = (int)bfi(0x80808080u, p1, ((p1 & 0x1f1f1f1fu) | cc[1]) + b4);
        bf[4 * hh + 2] = (int)bfi(0x80808080u, p2, ((p2 & 0x1f1f1f1fu) | cc[2]) + b4);
        const unsigned u = ((p1 >> 2) & 0x18181818u) | ((p0 >> 4) & 0x06060606u) | ((p2 >> 5) & 0x01010101u) | cc[3];
        bf[4 * hh + 3] = (int)bfi(0x80808080u, p2 << 1, u + b4);
      }
    }
    asm volatile("" ::: "memory");
    if (lane == 0) *vprog = (unsigned)(j + 1);
    acc0 = __builtin_amdgcn_mfma_scale_f32_32x32x64_f8f6f4(a0, bf, acc0, 0, 1, 0, 127, 0, 127);
    acc1 = __builtin_amdgcn_mfma_scale_f32_32x32x64_f8f6f4(a1, bf, acc1, 0, 1, 0, 127, 0, 127);
    so = so + SLOTB == RING * SLOTB ? 0 : so + SLOTB;
  }
  {
    const v16f give = pp ? acc0 : acc1;
#pragma unroll
    for (int q = 0; q < 4; ++q) {
      v4u v;
#pragma unroll
      for (int e = 0; e < 4; ++e) v[e] = __float_as_uint(give[4 * q + e]);
      *(LAS v4u*)(priv + q * 1024 + lane * 16) = v;
    }
    asm volatile("" ::: "memory");
    if (lane == 0) *(volatile LAS unsigned*)(ctrl + C_DONE + wave * 4) = 1u;
    while (!__builtin_amdgcn_readfirstlane((int)*(volatile LAS unsigned*)(ctrl + C_DONE + (wave ^ 4) * 4)))
      __builtin_amdgcn_s_sleep(1);
    asm volatile("" ::: "memory");
  }
  const bool failed = __builtin_amdgcn_readfirstlane((int)*vfail) != 0;
  LAS unsigned char* const other = smem + L_PRIV + (wave ^ 4) * 4096;
  v4u ov[4];
#pragma unroll
  for (int q = 0; q < 4; ++q) ov[q] = *(const LAS v4u*)(other + q * 1024 + lane * 16);
  float res[16];
#pragma unroll
  for (int r = 0; r < 16; ++r) {
    const float y = ((pp ? acc1[r] : acc0[r]) + __uint_as_float(ov[r >> 2][r & 3])) * scale;
    const float z = fminf(fmaxf(y, -60.0f), 60.0f);
    res[r] = failed ? __builtin_nanf("") : y * __builtin_amdgcn_rcpf(1.0f + __expf(-z));
  }
  asm volatile("" ::: "memory");
#pragma unroll
  for (int r = 0; r < 16; ++r)
    *(LAS float*)(other + ((r & 3) + 8 * (r >> 2) + 4 * g) * 128 + c * 4) = res[r];
  asm volatile("" ::: "memory");
  {
    const auto rsO = __builtin_amdgcn_make_buffer_rsrc((void*)p.out, 0, MS * NN * 4, RSRC3);
    const int oo = ((mblk * 64 + pp * 32 + (lane >> 3)) * NN + nblk * 128 + wc * 32 + (lane & 7) * 4) * 4;
#pragma unroll
    for (int q = 0; q < 4; ++q) {
      const v4u v = *(const LAS v4u*)(other + q * 1024 + lane * 16);
      __builtin_amdgcn_raw_buffer_store_b128(v, rsO, oo + q * 8 * NN * 4, 0, 16);
    }
  }
}

extern "C" void kernel_launch(int rank, int nranks, ncclComm_t comm,
                              hipStream_t stream,
                              void* const* d_in, const int* in_sizes, int n_in,
                              void* d_out, int out_size,
                              const int* in_dtypes, int out_dtype,
                              void* const* d_out_all,
                              void* const* d_ws_all, size_t ws_bytes) {
  (void)comm; (void)in_sizes; (void)n_in; (void)out_size; (void)in_dtypes; (void)out_dtype; (void)d_out_all;
  if (nranks != NR || (unsigned)rank >= (unsigned)NR || ws_bytes < WS_NEED) {
    fprintf(stderr, "kernel_launch: unexpected configuration\n");
    return;
  }
  Params p;
  p.x = (const uint8_t*)d_in[0];
  p.w = (const uint8_t*)d_in[1];
  p.sx = (const float*)d_in[2];
  p.sw = (const float*)d_in[3];
  p.out = (float*)d_out;
  for (int i = 0; i < NR; ++i) p.ws[i] = (uint8_t*)d_ws_all[i];
  p.rank = rank;
  hipLaunchKernelGGL(fused_ag_gemm, dim3(NWG), dim3(NTHREADS), 0, stream, p);
  HIP_CHECK(hipGetLastError());
}
```

```cpp
#include <hip/hip_runtime.h>
#include <rccl/rccl.h>
#include <cstdio>
#include <cstdint>

#define HIP_CHECK(expr) do { hipError_t _e = (expr); if (_e != hipSuccess) { \
    fprintf(stderr, "HIP error %s at %s:%d\n", hipGetErrorString(_e), __FILE__, __LINE__); \
    return; } } while (0)

typedef int      v8i  __attribute__((ext_vector_type(8)));
typedef unsigned v4u  __attribute__((ext_vector_type(4)));
typedef float    v16f __attribute__((ext_vector_type(16)));
#define LAS __attribute__((address_space(3)))

namespace {

constexpr int NR = 8;
constexpr int NN = 4096;
constexpr int MS = 512;
constexpr int KS = 512;
constexpr int NWG = 256;
constexpr int NTHREADS = 960;

constexpr size_t WFLAG_OFF = 0;
constexpr size_t XFLAG_OFF = 32768;
constexpr size_t TMO_OFF   = 40960;
constexpr size_t EPOCH_OFF = 49152;
constexpr size_t SLOT_OFF  = 1u << 20;
constexpr size_t WBLK      = 8192;
constexpr size_t WSRC      = 8 * 32 * WBLK;
constexpr size_t XSRC      = 32 * 512 * 16;
constexpr size_t X_OFF     = NR * WSRC;
constexpr size_t SLOT_BYTES = X_OFF + NR * XSRC;
constexpr size_t WS_NEED   = SLOT_OFF + 2 * SLOT_BYTES;

constexpr int RING = 5;
constexpr int SLOTB = 12288;
constexpr int FLY = 2;
constexpr int L_RING = 0;
constexpr int L_PRIV = 2 * RING * SLOTB;
constexpr int L_SLIST = L_PRIV + 8 * 4096;
constexpr int L_CTRL = L_SLIST + 528;
constexpr int C_ANY = 0;
constexpr int C_NSEQ = 8;
constexpr int C_FAIL = 12;
constexpr int C_RAWH = 16;
constexpr int C_PROG = 32;
constexpr int C_FULL = 64;
constexpr int C_DONE = 80;
constexpr int C_ANYRAW = 176;
constexpr int C_SEQ = 112;
constexpr int L_TOTAL = L_CTRL + 192;

constexpr unsigned long long DEADLINE_TICKS = 50000000ull;
constexpr int AUX_SYS = 17;
constexpr int RSRC3 = 0x00020000;
constexpr unsigned PACE = 116;
constexpr unsigned T_LEAD = 20;
constexpr unsigned XEARLY = 20;

struct Params {
  const uint8_t* x;
  const uint8_t* w;
  const float* sx;
  const float* sw;
  float* out;
  uint8_t* ws[NR];
  int rank;
};

#define WAIT_VM(n) asm volatile("s_waitcnt vmcnt(" #n ")" ::: "memory")

__device__ __forceinline__ v4u transpose4(unsigned a0, unsigned a1, unsigned a2, unsigned a3) {
  const unsigned t0 = __builtin_amdgcn_perm(a1, a0, 0x05010400u);
  const unsigned t1 = __builtin_amdgcn_perm(a1, a0, 0x07030602u);
  const unsigned t2 = __builtin_amdgcn_perm(a3, a2, 0x05010400u);
  const unsigned t3 = __builtin_amdgcn_perm(a3, a2, 0x07030602u);
  v4u r;
  r[0] = __builtin_amdgcn_perm(t2, t0, 0x05040100u);
  r[1] = __builtin_amdgcn_perm(t2, t0, 0x07060302u);
  r[2] = __builtin_amdgcn_perm(t3, t1, 0x05040100u);
  r[3] = __builtin_amdgcn_perm(t3, t1, 0x07060302u);
  return r;
}
__device__ __forceinline__ unsigned bfi(unsigned mask, unsigned a, unsigned b) { return (a & mask) | (b & ~mask); }
__device__ __forceinline__ unsigned long long uni64(unsigned lo, unsigned hi) {
  return ((unsigned long long)(unsigned)__builtin_amdgcn_readfirstlane(hi) << 32) |
         (unsigned)__builtin_amdgcn_readfirstlane(lo);
}

}

__global__ void __launch_bounds__(NTHREADS, 1) fused_ag_gemm(Params p) {
  __shared__ __attribute__((aligned(16))) unsigned char smem_[L_TOTAL];
  LAS unsigned char* const smem = (LAS unsigned char*)smem_;
  LAS unsigned char* const ctrl = smem + L_CTRL;
  const int b = blockIdx.x;
  const int wave = __builtin_amdgcn_readfirstlane(threadIdx.x >> 6);
  const int lane = threadIdx.x & 63;
  const int rank = p.rank;
  uint8_t* const myws = p.ws[rank];
  unsigned* const ep = (unsigned*)(myws + EPOCH_OFF) + b * 16 + wave;
  const unsigned epv = __hip_atomic_load(ep, __ATOMIC_RELAXED, __HIP_MEMORY_SCOPE_AGENT);
  unsigned* const tmo = (unsigned*)(myws + TMO_OFF);

  if (wave == 12 || wave == 13) {
    const int hw = wave - 12;
    const int a = lane & 15, kc = lane >> 4;
    const int dx = b >> 5, ix = b & 31;
    const auto rsW = __builtin_amdgcn_make_buffer_rsrc((void*)p.w, 0, KS * NN, RSRC3);
    const auto rsX = __builtin_amdgcn_make_buffer_rsrc((void*)p.x, 0, 4096 * KS, RSRC3);
    v4u xr[4];
    {
      const int off = (dx * MS + (ix >> 2) * 64 + lane) * KS + (ix & 3) * 128 + hw * 64;
#pragma unroll
      for (int t = 0; t < 4; ++t) xr[t] = __builtin_amdgcn_raw_buffer_load_b128(rsX, off + t * 16, 0, 0);
    }
    unsigned cur[16];
    {
      const int woff = ((b >> 5) * 64 + kc * 16) * NN + (b & 31) * 128 + hw * 64 + a * 4;
#pragma unroll
      for (int t = 0; t < 16; ++t) cur[t] = __builtin_amdgcn_raw_buffer_load_b32(rsW, woff + t * NN, 0, 0);
    }
    __builtin_amdgcn_s_barrier();
    __builtin_amdgcn_s_setprio(3);
    const unsigned call0 = __builtin_amdgcn_readfirstlane(epv);
    const unsigned call = call0 + 1;
    const unsigned par = call0 & 1;
    if (lane == 0) __hip_atomic_store(ep, call, __ATOMIC_RELAXED, __HIP_MEMORY_SCOPE_AGENT);
    asm volatile("s_waitcnt vmcnt(17)" : "+v"(xr[0]), "+v"(xr[1]), "+v"(xr[2]), "+v"(xr[3]) :: "memory");
    const unsigned long long tstart = __builtin_amdgcn_s_memrealtime() - T_LEAD;
    const size_t soff = SLOT_OFF + (size_t)par * SLOT_BYTES;
    const unsigned w64 = (unsigned)(ix * 2 + hw);
    auto wait_slot = [&](unsigned ticks) {
      for (;;) {
        const long long left = (long long)(tstart + ticks - __builtin_amdgcn_s_memrealtime());
        if (left <= 0) break;
        if (left > 40) __builtin_amdgcn_s_sleep(10); else __builtin_amdgcn_s_sleep(1);
      }
    };
    uint8_t* const dws = p.ws[dx];
    const bool remote = dx != rank;
    const auto rdX = __builtin_amdgcn_make_buffer_rsrc((void*)(dws + soff + X_OFF + (size_t)rank * XSRC), 0, (int)XSRC, RSRC3);
    const int xoff = (((ix & 3) * 2 + hw) * 8 + (ix >> 2)) * 4096 + lane * 16;
    unsigned* const xflag = (unsigned*)(dws + XFLAG_OFF) + par * 512 + ((ix >> 2) * 8 + rank) * 8 + (ix & 3) * 2 + hw;
    const unsigned xslot = (w64 * 4u * PACE) / 64u;
    const bool xearly = !remote || w64 < XEARLY;
    if (xearly) {
      if (remote) wait_slot(xslot);
#pragma unroll
      for (int t = 0; t < 4; ++t) __builtin_amdgcn_raw_buffer_store_b128(xr[t], rdX, xoff + t * 1024, 0, AUX_SYS);
    }

    v4u pk[4];
    unsigned cnt = 0;
    bool raw;
    {
      const unsigned rm = 0x00ff00ffu << (a & 8);
      const unsigned lt = (1u << (lane & 31)) - 1u;
      const unsigned below_lo = lane < 32 ? (rm & lt) : rm;
      const unsigned below_hi = lane < 32 ? 0u : (rm & lt);
      const unsigned posbase = (unsigned)(((kc & 1) << 10) | (((kc >> 1) * 32 + ((4 * a) & 31)) << 4));
      const unsigned lbase = (unsigned)(L_SLIST + (hw * 2 + (a >> 3)) * 128);
      unsigned s = 0;
#pragma unroll
      for (int t = 0; t < 16; t += 4) s = __builtin_amdgcn_sad_u8((cur[t] >> 2) & 0x1f1f1f1fu, 0u, s);
      int tot = 96;
#pragma unroll
      for (int bit = 2; bit < 9; ++bit) tot += __builtin_popcountll(__ballot((s >> bit) & 1u)) << bit;
      int base = (tot - 4291) >> 10;
      base = base < 0 ? 0 : (base > 24 ? 24 : base);
      const unsigned b4 = (unsigned)base * 0x04040404u;
      unsigned ap[16], ex[4];
#pragma unroll
      for (int t = 0; t < 16; ++t) {
        const unsigned dd = (cur[t] | 0x80808080u) - b4;
        const int tt = t & 3;
        const unsigned e2 = tt == 0 ? ((dd >> 5) & 0x03030303u) : tt == 1 ? ((dd >> 3) & 0x0c0c0c0cu)
                          : tt == 2 ? ((dd >> 1) & 0x30303030u) : ((dd << 1) & 0xc0c0c0c0u);
        ex[t >> 2] = tt == 0 ? e2 : (ex[t >> 2] | e2);
        ap[t] = bfi(0x80808080u, cur[t], dd) & 0x9f9f9f9fu;
      }
      unsigned nex = 0;
#pragma unroll
      for (int w = 0; w < 4; ++w) nex += __builtin_popcount((ex[w] | (ex[w] >> 1)) & 0x55555555u);
      const bool hopeless = __ballot(nex >= 16u) != 0ull;
#pragma unroll
      for (int w = 0; w < 4; ++w) {
        unsigned e = hopeless ? 0u : ex[w];
        for (;;) {
          const unsigned long long m = __ballot(e != 0u);
          if (!m) break;
          const unsigned mlo = (unsigned)m, mhi = (unsigned)(m >> 32);
          const unsigned slotn = cnt + 1u + __builtin_popcount(mlo & below_lo) + __builtin_popcount(mhi & below_hi);
          const unsigned bb = (unsigned)__builtin_ctz(e | 0x80000000u) & 30u;
          const unsigned hc = (e >> bb) & 3u;
          const unsigned pos = (unsigned)(4 * w) + posbase + ((bb & 0x18u) << 1) + ((bb >> 1) & 3u);
          const unsigned la = (e != 0u && slotn <= 63u) ? lbase + 2u * slotn : (unsigned)(L_SLIST + 512);
          *(LAS unsigned short*)(smem + la) = (unsigned short)((pos << 2) | hc);
          e ^= hc << bb;
          cnt += __builtin_popcount(mlo & rm) + __builtin_popcount(mhi & rm);
        }
      }
      raw = hopeless || __ballot(cnt > 63u) != 0ull;
      if (raw) {
#pragma unroll
        for (int q = 0; q < 4; ++q) pk[q] = transpose4(cur[4 * q], cur[4 * q + 1], cur[4 * q + 2], cur[4 * q + 3]);
      } else {
        if ((lane & 55) == 0) *(LAS unsigned short*)(smem + lbase) = (unsigned short)(cnt | ((unsigned)base << 6));
#pragma unroll
        for (int r = 0; r < 4; ++r) {
          const unsigned v = ap[12 + r];
          ap[0 + r] |= (v << 4) & 0x60606060u;
          ap[4 + r] |= (v << 2) & 0x60606060u;
          ap[8 + r] |= ((v >> 1) & 0x40404040u) | ((v << 5) & 0x20202020u);
        }
#pragma unroll
        for (int q = 0; q < 3; ++q) pk[q] = transpose4(ap[4 * q], ap[4 * q + 1], ap[4 * q + 2], ap[4 * q + 3]);
        asm volatile("" ::: "memory");
        pk[3] = *(LAS v4u*)(smem + L_SLIST + hw * 256 + a * 16);
      }
    }
    const auto rlW = __builtin_amdgcn_make_buffer_rsrc((void*)(myws + soff + (size_t)rank * WSRC), 0, (int)WSRC, RSRC3);
    const auto rdW = __builtin_amdgcn_make_buffer_rsrc((void*)(dws + soff + (size_t)rank * WSRC), 0, (int)WSRC, RSRC3);
    unsigned* const lflags = (unsigned*)(myws + WFLAG_OFF) + par * 4096 + rank * 16;
    unsigned* const dflags = (unsigned*)(dws + WFLAG_OFF) + par * 4096 + rank * 16;
    auto fidx = [&](unsigned it) { return ((it >> 1) & 31u) * 128u + (it >> 6) * 2u + (it & 1u); };
    auto off0 = [&](unsigned it, bool r) { return (int)(it >> 1) * (int)WBLK + kc * (r ? 2048 : 1536) + (int)(it & 1u) * 256 + a * 16; };
    auto off3 = [&](unsigned it, bool r) {
      return r ? off0(it, true) + 1536 : (int)(it >> 1) * (int)WBLK + 6144 + (int)(it & 1u) * 256 + a * 16; };
    unsigned item = (unsigned)b * 2u + (unsigned)hw;
    unsigned fv = call * 2 + (raw ? 1u : 0u);
    {
      const int db = off0(item, raw), db3 = off3(item, raw);
#pragma unroll
      for (int q = 0; q < 3; ++q) __builtin_amdgcn_raw_buffer_store_b128(pk[q], rlW, db + q * 512, 0, AUX_SYS);
      if (raw || (lane < 16 && (unsigned)(lane & 7) * 8u <= cnt))
        __builtin_amdgcn_raw_buffer_store_b128(pk[3], rlW, db3, 0, AUX_SYS);
      WAIT_VM(0);
      if (lane == 0) {
        __hip_atomic_store(lflags + fidx(item), fv, __ATOMIC_RELAXED, __HIP_MEMORY_SCOPE_SYSTEM);
        if (xearly) __hip_atomic_store(xflag, call, __ATOMIC_RELAXED, __HIP_MEMORY_SCOPE_SYSTEM);
      }
    }
    if (!xearly) {
      wait_slot(xslot);
#pragma unroll
      for (int t = 0; t < 4; ++t) __builtin_amdgcn_raw_buffer_store_b128(xr[t], rdX, xoff + t * 1024, 0, AUX_SYS);
      WAIT_VM(0);
      if (lane == 0) __hip_atomic_store(xflag, call, __ATOMIC_RELAXED, __HIP_MEMORY_SCOPE_SYSTEM);
    }
    if (remote) {
      const unsigned col = (w64 + (unsigned)rank * 8u) & 63u;
      unsigned slot_t = 4u * PACE + (w64 * (16u * PACE / 5u)) / 64u;
      const unsigned litem = (unsigned)((lane & 7) < dx ? (lane & 7) : (lane & 7) + 1) * 64u + col;
      const unsigned* const lf = lflags + fidx(litem & 511u);
      unsigned lfv = 0;
#pragma unroll 1
      for (int n = 0; n < 8; ++n) {
        const bool r = fv & 1u;
        const int db = off0(item, r), db3 = off3(item, r);
        const unsigned lc = (unsigned)__shfl((int)pk[3][0], lane & 8) & 63u;
        unsigned nitem = 0, nfv = 0;
        v4u n0 = pk[0], n1 = pk[1], n2 = pk[2], n3 = pk[3];
        if (n < 7) {
          nitem = (unsigned)(n < dx ? n : n + 1) * 64u + col;
          while ((int)((nfv = __builtin_amdgcn_readlane(lfv, n)) - 2u * call) < 0) {
            lfv = __hip_atomic_load(lf, __ATOMIC_RELAXED, __HIP_MEMORY_SCOPE_AGENT);
            if ((int)(__builtin_amdgcn_readlane(lfv, n) - 2u * call) >= 0) continue;
            if (__builtin_amdgcn_s_memrealtime() - tstart > DEADLINE_TICKS) {
              if (lane == 0) __hip_atomic_store(tmo, call, __ATOMIC_RELAXED, __HIP_MEMORY_SCOPE_AGENT);
              nfv = 0;
              break;
            }
            __builtin_amdgcn_s_sleep(4);
          }
          asm volatile("" ::: "memory");
          const bool nr = nfv & 1u;
          const int nb0 = off0(nitem, nr), nb3 = off3(nitem, nr);
          n0 = __builtin_amdgcn_raw_buffer_load_b128(rlW, nb0, 0, 0);
          n1 = __builtin_amdgcn_raw_buffer_load_b128(rlW, nb0 + 512, 0, 0);
          n2 = __builtin_amdgcn_raw_buffer_load_b128(rlW, nb0 + 1024, 0, 0);
          n3 = __builtin_amdgcn_raw_buffer_load_b128(rlW, nb3, 0, 0);
        }
        wait_slot(slot_t);
        slot_t += r ? 4u * PACE : 16u * PACE / 5u;
        __builtin_amdgcn_raw_buffer_store_b128(pk[0], rdW, db, 0, AUX_SYS);
        __builtin_amdgcn_raw_buffer_store_b128(pk[1], rdW, db + 512, 0, AUX_SYS);
        WAIT_VM(1);
        __builtin_amdgcn_raw_buffer_store_b128(pk[2], rdW, db + 1024, 0, AUX_SYS);
        if (r) WAIT_VM(1);
        if (r || (lane < 16 && (unsigned)(lane & 7) * 8u <= lc))
          __builtin_amdgcn_raw_buffer_store_b128(pk[3], rdW, db3, 0, AUX_SYS);
        WAIT_VM(0);
        if (lane == 0 && fv) __hip_atomic_store(dflags + fidx(item), fv, __ATOMIC_RELAXED, __HIP_MEMORY_SCOPE_SYSTEM);
        pk[0] = n0; pk[1] = n1; pk[2] = n2; pk[3] = n3;
        item = nitem;
        fv = nfv;
      }
    }
    return;
  }

  if (wave == 14 && lane < 12) *(LAS v4u*)(ctrl + lane * 16) = v4u{0u, 0u, 0u, 0u};
  asm volatile("s_waitcnt lgkmcnt(0)" ::: "memory");
  __builtin_amdgcn_s_barrier();

  const unsigned call0 = __builtin_amdgcn_readfirstlane(epv);
  const unsigned call = call0 + 1;
  const unsigned par = call0 & 1;
  if (lane == 0) __hip_atomic_store(ep, call, __ATOMIC_RELAXED, __HIP_MEMORY_SCOPE_AGENT);
  const int nblk = (b & 7) * 4 + ((b >> 3) & 3);
  const int mblk = b >> 5;
  volatile LAS unsigned* const vfail = (volatile LAS unsigned*)(ctrl + C_FAIL);

  if (wave == 14) {
    __builtin_amdgcn_s_setprio(2);
    const unsigned long long* const wfl = (const unsigned long long*)(myws + WFLAG_OFF) + par * 2048 + nblk * 64 + lane;
    const unsigned* const xfl = (const unsigned*)(myws + XFLAG_OFF) + par * 512 + mblk * 64 + lane;
    unsigned long long xok = 0, seen = 0, raw0 = 0, raw1 = 0;
    int nseq = 0;
    const unsigned long long t0 = __builtin_amdgcn_s_memrealtime();
    for (;;) {
      const unsigned long long wf = __hip_atomic_load(wfl, __ATOMIC_RELAXED, __HIP_MEMORY_SCOPE_SYSTEM);
      if (~xok) {
        const unsigned xf = __hip_atomic_load(xfl, __ATOMIC_RELAXED, __HIP_MEMORY_SCOPE_SYSTEM);
        xok = __ballot((int)(xf - call) >= 0);
      }
      const unsigned w0 = (unsigned)wf, w1 = (unsigned)(wf >> 32);
      unsigned long long rdy = __ballot((int)(w0 - 2u * call) >= 0 && (int)(w1 - 2u * call) >= 0) & xok;
      const unsigned long long fresh = rdy & ~seen;
      if (fresh) {
        seen = rdy;
        const int pos = nseq + __builtin_popcountll(fresh & ((1ull << lane) - 1ull));
        if ((fresh >> lane) & 1ull) *(volatile LAS unsigned char*)(ctrl + C_SEQ + pos) = (unsigned char)lane;
        const unsigned long long m0 = __ballot(w0 & 1u), m1 = __ballot(w1 & 1u);
        unsigned long long rw = (m0 | m1) & fresh;
        if (rw) {
          while (rw) {
            const int st = __builtin_ctzll(rw);
            rw &= rw - 1;
            const int ps = nseq + __builtin_popcountll(fresh & ((1ull << st) - 1ull));
            raw0 |= ((m0 >> st) & 1ull) << ps;
            raw1 |= ((m1 >> st) & 1ull) << ps;
          }
          if (lane == 0) {
            *(volatile LAS unsigned long long*)(ctrl + C_RAWH) = raw0;
            *(volatile LAS unsigned long long*)(ctrl + C_RAWH + 8) = raw1;
            *(volatile LAS unsigned long long*)(ctrl + C_ANY) = raw0 | raw1;
            *(volatile LAS unsigned*)(ctrl + C_ANYRAW) = 1u;
          }
        }
        nseq += __builtin_popcountll(fresh);
        if (lane == 0) *(volatile LAS unsigned*)(ctrl + C_NSEQ) = (unsigned)nseq;
        if (!~rdy) break;
        continue;
      }
      const unsigned long long dt = __builtin_amdgcn_s_memrealtime() - t0;
      if (dt > 20000) {
        if (__builtin_amdgcn_readfirstlane(__hip_atomic_load(tmo, __ATOMIC_RELAXED, __HIP_MEMORY_SCOPE_AGENT)) == call ||
            dt > DEADLINE_TICKS) {
          if (lane == 0) {
            __hip_atomic_store(tmo, call, __ATOMIC_RELAXED, __HIP_MEMORY_SCOPE_AGENT);
            *vfail = 1;
          }
          break;
        }
        __builtin_amdgcn_s_sleep(127);
        __builtin_amdgcn_s_sleep(127);
      } else {
        __builtin_amdgcn_s_sleep(4);
      }
    }
    return;
  }

  uint8_t* const slot = myws + SLOT_OFF + (size_t)par * SLOT_BYTES;

  if (wave >= 8) {
    __builtin_amdgcn_s_setprio(2);
    const int L = (wave - 8) >> 1, half = wave & 1;
    const unsigned ring_addr = (unsigned)(size_t)(smem + L_RING) + (unsigned)L * (RING * SLOTB);
    const unsigned vo = (unsigned)lane * 16u;
    const uint8_t* const xsrc = slot + X_OFF + (size_t)mblk * 4096;
    const uint8_t* const wsrc = slot + (size_t)nblk * WBLK;
    volatile LAS unsigned* const vfull = (volatile LAS unsigned*)(ctrl + C_FULL + (wave - 8) * 4);
    unsigned long long posraw = 0;
    int seqv = 0;
    int j = half, avail = 0, nis = 0, full = 0;
    unsigned dst = ring_addr + (unsigned)half * SLOTB;
    WAIT_VM(0);
    while (full < 16) {
      const v4u ct = *(volatile LAS v4u*)(ctrl + C_ANY);
      const v4u pr = *(volatile LAS v4u*)(ctrl + C_PROG + L * 16);
      const int sv = *(volatile LAS unsigned char*)(ctrl + C_SEQ + lane);
      const bool go = j < avail;
      if (go) {
        const int pos = 2 * j + L;
        const int st = __builtin_amdgcn_readlane(seqv, pos);
        const uint8_t* const xa = xsrc + (size_t)st * 32768;
        const uint8_t* const wa = wsrc + (size_t)st * (32 * WBLK);
        const uint8_t* const wb = wa + 4096;
        unsigned keep;
        asm volatile(
            "s_nop 4\n\t"
            "s_mov_b32 %0, m0\n\t"
            "s_mov_b32 m0, %5\n\t"
            "s_nop 0\n\t"
            "global_load_lds_dwordx4 %1, %2\n\t"
            "global_load_lds_dwordx4 %1, %2 offset:1024\n\t"
            "global_load_lds_dwordx4 %1, %2 offset:2048\n\t"
            "global_load_lds_dwordx4 %1, %2 offset:3072\n\t"
            "s_add_u32 m0, m0, 0x1000\n\t"
            "s_nop 0\n\t"
            "global_load_lds_dwordx4 %1, %3\n\t"
            "global_load_lds_dwordx4 %1, %3 offset:1024\n\t"
            "global_load_lds_dwordx4 %1, %3 offset:2048\n\t"
            "global_load_lds_dwordx4 %1, %3 offset:3072\n\t"
            "s_add_u32 m0, m0, 0x1000\n\t"
            "s_nop 0\n\t"
            "global_load_lds_dwordx4 %1, %4\n\t"
            "global_load_lds_dwordx4 %1, %4 offset:1024\n\t"
            "global_load_lds_dwordx4 %1, %4 offset:2048\n\t"
            "s_mov_b32 m0, %0"
            : "=&s"(keep)
            : "v"(vo), "s"(xa), "s"(wa), "s"(wb), "s"(dst)
            : "memory", "scc");
        if ((posraw >> pos) & 1ull) {
          asm volatile(
              "s_mov_b32 %0, m0\n\t"
              "s_mov_b32 m0, %3\n\t"
              "s_nop 0\n\t"
              "global_load_lds_dwordx4 %1, %2 offset:3072\n\t"
              "s_mov_b32 m0, %0"
              : "=&s"(keep)
              : "v"(vo), "s"(wb), "s"(dst + 8192u)
              : "memory");
        }
        j += 2;
        ++nis;
        dst += 2 * SLOTB;
        if (dst >= ring_addr + RING * SLOTB) dst -= RING * SLOTB;
        if (nis - full == FLY) {
          WAIT_VM(11);
          ++full;
          if (lane == 0) *vfull = (unsigned)full;
        }
      }
      seqv = sv;
      const int lim = (int)min(min(pr[0], pr[1]), min(pr[2], pr[3])) + RING;
      avail = __builtin_amdgcn_readfirstlane(min(lim, ((int)ct[2] + 1 - L) >> 1));
      posraw = uni64(ct[0], ct[1]);
      if (go || j < avail) continue;
      if (nis > full) {
        WAIT_VM(0);
        full = nis;
        if (lane == 0) *vfull = (unsigned)full;
        continue;
      }
      if (__builtin_amdgcn_readfirstlane((int)ct[3])) { if (lane == 0) *vfull = 16u; break; }
      __builtin_amdgcn_s_sleep(1);
    }
    return;
  }

  const int wc = wave & 3, pp = wave >> 2;
  const int g = lane >> 5, c = lane & 31;
  LAS unsigned char* const priv = smem + L_PRIV + wave * 4096;
  *(LAS v4u*)(priv + lane * 16) = v4u{0u, 0u, 0u, 0u};
  *(LAS v4u*)(priv + 1024 + lane * 16) = v4u{0u, 0u, 0u, 0u};
  const float scale = p.sx[0] * p.sw[0];
  const LAS unsigned char* const ring = smem + L_RING + pp * (RING * SLOTB);
  const int aoff = (2 * g) * 1024 + c * 16;
  const int roff = 4096 + (8 * g) * 512 + wc * 128 + c * 4;
  const int poff = 4096 + (6 * g) * 512 + wc * 128 + c * 4;
  const int loff = 4096 + 12 * 512 + wc * 128 + lane * 2;
  volatile LAS unsigned* const vprog = (volatile LAS unsigned*)(ctrl + C_PROG + wave * 4);

  v16f acc0 = {}, acc1 = {};
  int nfull = 0;
  unsigned long long posraw = 0;
  int so = 0;
  for (int j = 0; j < 32; ++j) {
    if (nfull <= j) {
      for (;;) {
        const unsigned long long f2 = *(volatile LAS unsigned long long*)(ctrl + C_FULL + pp * 8);
        const unsigned anyraw = *(volatile LAS unsigned*)(ctrl + C_ANYRAW);
        nfull = __builtin_amdgcn_readfirstlane((int)min(2u * (unsigned)f2, 2u * (unsigned)(f2 >> 32) + 1u));
        if (nfull > j) {
          if (__builtin_amdgcn_readfirstlane((int)anyraw)) {
            const unsigned long long r = *(volatile LAS unsigned long long*)(ctrl + C_RAWH + (wc >> 1) * 8);
            posraw = uni64((unsigned)r, (unsigned)(r >> 32)) >> pp;
          }
          break;
        }
        __builtin_amdgcn_s_sleep(1);
      }
    }
    asm volatile("" ::: "memory");
    const LAS unsigned char* const sl = ring + so;
    v8i a0, a1, bf;
    const unsigned ent = *(const LAS unsigned short*)(sl + loff);
#pragma unroll
    for (int hh = 0; hh < 2; ++hh) {
      const v4u t0v = *(const LAS v4u*)(sl + aoff + hh * 1024);
      const v4u t1v = *(const LAS v4u*)(sl + aoff + hh * 1024 + 512);
#pragma unroll
      for (int e = 0; e < 4; ++e) { a0[hh * 4 + e] = (int)t0v[e]; a1[hh * 4 + e] = (int)t1v[e]; }
    }
    __builtin_amdgcn_sched_barrier(0);
    if ((posraw >> (2 * j)) & 1ull) {
#pragma unroll
      for (int q = 0; q < 8; ++q) bf[q] = *(const LAS int*)(sl + roff + q * 512);
    } else {
      unsigned pa[6];
#pragma unroll
      for (int q = 0; q < 6; ++q) pa[q] = *(const LAS unsigned*)(sl + poff + q * 512);
      const unsigned hdr = __builtin_amdgcn_readfirstlane(ent);
      const unsigned b4 = ((hdr >> 6) & 31u) * 0x04040404u;
      const unsigned ca = (unsigned)(lane - 1) < (hdr & 63u) ? ((ent >> 2) & 2047u) : 2048u + (unsigned)lane * 4u;
      priv[ca] = (unsigned char)((ent & 3u) << 5);
      const v4u c0 = *(const LAS v4u*)(priv + lane * 16);
      const v4u c1 = *(const LAS v4u*)(priv + 1024 + lane * 16);
      priv[ca] = 0;
#pragma unroll
      for (int hh = 0; hh < 2; ++hh) {
        const v4u cc = hh ? c1 : c0;
        const unsigned p0 = pa[3 * hh], p1 = pa[3 * hh + 1], p2 = pa[3 * hh + 2];
        bf[4 * hh + 0] = (int)bfi(0x80808080u, p0, ((p0 & 0x1f1f1f1fu) | cc[0]) + b4);
        bf[4 * hh + 1] = (int)bfi(0x80808080u, p1, ((p1 & 0x1f1f1f1fu) | cc[1]) + b4);
        bf[4 * hh + 2] = (int)bfi(0x80808080u, p2, ((p2 & 0x1f1f1f1fu) | cc[2]) + b4);
        const unsigned u = ((p1 >> 2) & 0x18181818u) | ((p0 >> 4) & 0x06060606u) | ((p2 >> 5) & 0x01010101u) | cc[3];
        bf[4 * hh + 3] = (int)bfi(0x80808080u, p2 << 1, u + b4);
      }
    }
    asm volatile("" ::: "memory");
    if (lane == 0) *vprog = (unsigned)(j + 1);
    acc0 = __builtin_amdgcn_mfma_scale_f32_32x32x64_f8f6f4(a0, bf, acc0, 0, 1, 0, 127, 0, 127);
    acc1 = __builtin_amdgcn_mfma_scale_f32_32x32x64_f8f6f4(a1, bf, acc1, 0, 1, 0, 127, 0, 127);
    so = so + SLOTB == RING * SLOTB ? 0 : so + SLOTB;
  }
  {
    const v16f give = pp ? acc0 : acc1;
#pragma unroll
    for (int q = 0; q < 4; ++q) {
      v4u v;
#pragma unroll
      for (int e = 0; e < 4; ++e) v[e] = __float_as_uint(give[4 * q + e]);
      *(LAS v4u*)(priv + q * 1024 + lane * 16) = v;
    }
    asm volatile("" ::: "memory");
    if (lane == 0) *(volatile LAS unsigned*)(ctrl + C_DONE + wave * 4) = 1u;
    while (!__builtin_amdgcn_readfirstlane((int)*(volatile LAS unsigned*)(ctrl + C_DONE + (wave ^ 4) * 4)))
      __builtin_amdgcn_s_sleep(1);
    asm volatile("" ::: "memory");
  }
  const bool failed = __builtin_amdgcn_readfirstlane((int)*vfail) != 0;
  LAS unsigned char* const other = smem + L_PRIV + (wave ^ 4) * 4096;
  v4u ov[4];
#pragma unroll
  for (int q = 0; q < 4; ++q) ov[q] = *(const LAS v4u*)(other + q * 1024 + lane * 16);
  float res[16];
#pragma unroll
  for (int r = 0; r < 16; ++r) {
    const float y = ((pp ? acc1[r] : acc0[r]) + __uint_as_float(ov[r >> 2][r & 3])) * scale;
    const float z = fminf(fmaxf(y, -60.0f), 60.0f);
    res[r] = failed ? __builtin_nanf("") : y * __builtin_amdgcn_rcpf(1.0f + __expf(-z));
  }
  asm volatile("" ::: "memory");
#pragma unroll
  for (int r = 0; r < 16; ++r)
    *(LAS float*)(other + ((r & 3) + 8 * (r >> 2) + 4 * g) * 128 + c * 4) = res[r];
  asm volatile("" ::: "memory");
  {
    const auto rsO = __builtin_amdgcn_make_buffer_rsrc((void*)p.out, 0, MS * NN * 4, RSRC3);
    const int oo = ((mblk * 64 + pp * 32 + (lane >> 3)) * NN + nblk * 128 + wc * 32 + (lane & 7) * 4) * 4;
#pragma unroll
    for (int q = 0; q < 4; ++q) {
      const v4u v = *(const LAS v4u*)(other + q * 1024 + lane * 16);
      __builtin_amdgcn_raw_buffer_store_b128(v, rsO, oo + q * 8 * NN * 4, 0, 16);
    }
  }
}

extern "C" void kernel_launch(int rank, int nranks, ncclComm_t comm,
                              hipStream_t stream,
                              void* const* d_in, const int* in_sizes, int n_in,
                              void* d_out, int out_size,
                              const int* in_dtypes, int out_dtype,
                              void* const* d_out_all,
                              void* const* d_ws_all, size_t ws_bytes) {
  (void)comm; (void)in_sizes; (void)n_in; (void)out_size; (void)in_dtypes; (void)out_dtype; (void)d_out_all;
  if (nranks != NR || (unsigned)rank >= (unsigned)NR || ws_bytes < WS_NEED) {
    fprintf(stderr, "kernel_launch: unexpected configuration\n");
    return;
  }
  Params p;
  p.x = (const uint8_t*)d_in[0];
  p.w = (const uint8_t*)d_in[1];
  p.sx = (const float*)d_in[2];
  p.sw = (const float*)d_in[3];
  p.out = (float*)d_out;
  for (int i = 0; i < NR; ++i) p.ws[i] = (uint8_t*)d_ws_all[i];
  p.rank = rank;
  hipLaunchKernelGGL(fused_ag_gemm, dim3(NWG), dim3(NTHREADS), 0, stream, p);
  HIP_CHECK(hipGetLastError());
}
```
